```python
import numpy as np

kernel_name = "gpu_ag_gemm_m4096_k4096_n16384_bf16_reluq8"


ABSMAX_RTOL = 0.02

COMPUTE_DTYPE = "bfloat16"
EPILOGUE = "relu_quant_int8"
IN_QUANT = None
OUT_QUANT = "int8"
IN_DTYPES = {"x": "bfloat16", "w_mat": "bfloat16"}
WORLD_SIZE = 8

def _to_bf16(x):
    u = np.ascontiguousarray(x, dtype=np.float32).view(np.uint32)
    return ((u + 0x7FFF + ((u >> 16) & 1)) & 0xFFFF0000).view(np.float32)

def setup_inputs():

    x = _to_bf16(np.random.randn(4096, 4096).astype(np.float32))
    w_mat = _to_bf16((np.random.randn(4096, 16384) / np.sqrt(4096)).astype(np.float32))
    return {"x": x, "w_mat": w_mat}

def _quant_dequant(y, amax):
    scale = amax / 127.0
    q = np.clip(np.round(y / scale), -127.0, 127.0)
    return (q * scale).astype(np.float32)
def _epilogue(y):
    y = np.maximum(y, 0.0)
    return _quant_dequant(y, np.abs(y).max())

def reference(x, w_mat):
    y = (x @ w_mat).astype(np.float32)
    return _epilogue(y)
```

```cpp
#include <hip/hip_runtime.h>
#include <rccl/rccl.h>
#include <cstdio>

#define HIP_CHECK(expr) do { hipError_t _e = (expr); if (_e != hipSuccess) { \
    fprintf(stderr, "HIP error %s at %s:%d\n", hipGetErrorString(_e), __FILE__, __LINE__); \
    return; } } while (0)

typedef unsigned u32;
typedef unsigned long long u64;
typedef u32 u32x4 __attribute__((ext_vector_type(4)));
typedef u32 u32x2 __attribute__((ext_vector_type(2)));
typedef unsigned short u16x2 __attribute__((ext_vector_type(2)));
typedef float f32x32 __attribute__((ext_vector_type(32)));
#define LAS __attribute__((address_space(3)))
#define GAS __attribute__((address_space(1)))

constexpr int NRANK = 8;
constexpr int GM = 4096, GK = 4096, NL = 2048, MS = 512;
constexpr int NWG = 256;
constexpr int NGRP = 32;
constexpr int NGEMM_PER = 26;
constexpr int NCOMM_PER = 6;
#define RAWKB 4
constexpr int NRAW = 8 * RAWKB;
constexpr int G0 = RAWKB / 4;
constexpr int NQUAD = 8 * (NGRP - G0);
constexpr u64 DEADLINE = 40000000ull;
constexpr u32 SPIN_CAP = 1u << 24;

constexpr size_t OFF_CNT   = 0;
constexpr size_t OFF_TMO   = 32768;
constexpr size_t OFF_AMAXG = 98304;
constexpr size_t OFF_AMAXL = 65536;
constexpr size_t OFF_RFLAG = 262144;
constexpr size_t OFF_RAWF  = 786432;
constexpr size_t OFF_XFLAG = 1u << 20;
constexpr size_t OFF_PK    = 2u << 20;
constexpr u32    PK_STRIDE = 3072;
constexpr size_t OFF_XG    = 6u << 20;
constexpr u32    XPITCH    = 260 * 1024;
constexpr size_t XG_BYTES  = (size_t)128 * XPITCH;
constexpr size_t WS_NEED   = OFF_XG + XG_BYTES + 4 * XPITCH;

constexpr int L_PAIR = 0;
constexpr int L_ABORT = 64, L_MAX = 68, L_MCNT = 72, L_ECNT = 76, L_LAND = 80, L_RDY = 84, L_TICK = 88, L_ECNT2 = 92, L_RES = 96;
constexpr int L_FREE = 128;
constexpr int L_CONS = 1024;
constexpr int L_LUT = 2048;
constexpr int L_RING = 4096, RING = 18, SLOT = 8192;
constexpr int L_ESCR = L_RING + RING * SLOT, ESCR = 2304;
constexpr int LDS_BYTES = L_ESCR + 4 * ESCR;
constexpr int CRING = 32;
constexpr int L_STRM = L_RING + CRING * 3072, STRM = 1152;

struct Params { const char* x; const char* w; char* out; int rank; int pad; char* rot[NRANK]; };

template <int N> __device__ __forceinline__ void waitvm() { asm volatile("s_waitcnt vmcnt(%0)" :: "n"(N) : "memory"); }
__device__ __forceinline__ u64 rt() { return __builtin_amdgcn_s_memrealtime(); }
__device__ __forceinline__ u32 rfl(u32 v) { return __builtin_amdgcn_readfirstlane(v); }

__device__ __forceinline__ u32 lds_ld(u32 addr) {
    u32 v; asm volatile("ds_read_b32 %0, %1\n\ts_waitcnt lgkmcnt(0)" : "=&v"(v) : "v"(addr) : "memory"); return v; }
__device__ __forceinline__ void lds_st(u32 addr, u32 v) { asm volatile("ds_write_b32 %0, %1" :: "v"(addr), "v"(v) : "memory"); }

__device__ __forceinline__ u32 ld_agent(const char* p) { return __hip_atomic_load((GAS u32*)p, __ATOMIC_RELAXED, __HIP_MEMORY_SCOPE_AGENT); }
__device__ __forceinline__ u32 ld_sys(const char* p) { return __hip_atomic_load((GAS u32*)p, __ATOMIC_RELAXED, __HIP_MEMORY_SCOPE_SYSTEM); }
__device__ __forceinline__ u64 ld_sys64(const char* p) { return __hip_atomic_load((GAS u64*)p, __ATOMIC_RELAXED, __HIP_MEMORY_SCOPE_SYSTEM); }
__device__ __forceinline__ u32 poll32(const char* p) {
    u32 v; asm volatile("global_atomic_add %0, %1, %2, off sc0 sc1\n\ts_waitcnt vmcnt(0)" : "=&v"(v) : "v"(p), "v"(0u) : "memory"); return v; }
__device__ __forceinline__ void st_agent(char* p, u32 v) { __hip_atomic_store((GAS u32*)p, v, __ATOMIC_RELAXED, __HIP_MEMORY_SCOPE_AGENT); }
__device__ __forceinline__ void st_sys64(char* p, u64 v) { __hip_atomic_store((GAS u64*)p, v, __ATOMIC_RELAXED, __HIP_MEMORY_SCOPE_SYSTEM); }

__device__ __forceinline__ void gst4_sys(char* p, u32x4 v) {
    asm volatile("global_store_dwordx4 %0, %1, off sc0 sc1\n\ts_nop 1" :: "v"(p), "v"(v) : "memory"); }
__device__ __forceinline__ void gst1_sys(char* p, u32 v) {
    asm volatile("global_store_dword %0, %1, off sc0 sc1" :: "v"(p), "v"(v) : "memory"); }
__device__ __forceinline__ void gst4_agent(char* p, u32x4 v) {
    asm volatile("global_store_dwordx4 %0, %1, off sc1\n\ts_nop 1" :: "v"(p), "v"(v) : "memory"); }
__device__ __forceinline__ void publish(char* flag, u32 v, u32 lane) { waitvm<0>(); if (lane == 0) st_agent(flag, v); }
__device__ __forceinline__ u32x4 gld4_sys_wait(const char* p) {
    u32x4 v; asm volatile("global_load_dwordx4 %0, %1, off sc0 sc1\n\ts_waitcnt vmcnt(0)" : "=&v"(v) : "v"(p) : "memory"); return v; }

__device__ __forceinline__ u32 bfi(u32 m, u32 a, u32 b) { return (m & a) | (~m & b); }
__device__ __forceinline__ u32 perm(u32 a, u32 b, u32 s) { return __builtin_amdgcn_perm(a, b, s); }
__device__ __forceinline__ u32 wave_scan(u32 v) {
    v += (u32)__builtin_amdgcn_update_dpp(0, (int)v, 0x111, 0xf, 0xf, false);
    v += (u32)__builtin_amdgcn_update_dpp(0, (int)v, 0x112, 0xf, 0xf, false);
    v += (u32)__builtin_amdgcn_update_dpp(0, (int)v, 0x114, 0xf, 0xf, false);
    v += (u32)__builtin_amdgcn_update_dpp(0, (int)v, 0x118, 0xf, 0xf, false);
    v += (u32)__builtin_amdgcn_update_dpp(0, (int)v, 0x142, 0xa, 0xf, false);
    v += (u32)__builtin_amdgcn_update_dpp(0, (int)v, 0x143, 0xc, 0xf, false);
    return v;
}
__device__ __forceinline__ u32 wave_maxu(u32 v) {
    v = max(v, (u32)__builtin_amdgcn_update_dpp(0, (int)v, 0x111, 0xf, 0xf, false));
    v = max(v, (u32)__builtin_amdgcn_update_dpp(0, (int)v, 0x112, 0xf, 0xf, false));
    v = max(v, (u32)__builtin_amdgcn_update_dpp(0, (int)v, 0x114, 0xf, 0xf, false));
    v = max(v, (u32)__builtin_amdgcn_update_dpp(0, (int)v, 0x118, 0xf, 0xf, false));
    v = max(v, (u32)__builtin_amdgcn_update_dpp(0, (int)v, 0x142, 0xa, 0xf, false));
    v = max(v, (u32)__builtin_amdgcn_update_dpp(0, (int)v, 0x143, 0xc, 0xf, false));
    return (u32)__builtin_amdgcn_readlane((int)v, 63);
}
__device__ __forceinline__ u32 rot8x4(u32 e) { return ((e >> 1) & 0x7f7f7f7fu) | ((e << 7) & 0x80808080u); }

constexpr u32 NPATCH = 31;
__device__ __forceinline__ u32 ref_exponent(const u32x4& f) {
    u32 s2 = 0;
#pragma unroll
    for (int i = 0; i < 4; ++i) s2 += (f[i] >> 7) & 0x00ff00ffu;
    const u32 tot = (u32)__builtin_amdgcn_readlane((int)wave_scan((s2 & 0xffffu) + (s2 >> 16)), 63);
    return min(max(((tot + 491u) >> 9) + 2u, 10u), 255u);
}
__device__ __forceinline__ void encode_block(const u32x4 (&f)[4], u32 B1, u32 lane, u32 scr, char* pk16, char* pk8) {
    u32 d[16];
#pragma unroll
    for (int i = 0; i < 16; ++i) d[i] = f[i >> 2][i & 3];
    const u16x2 B2 = {(unsigned short)(B1 - 1), (unsigned short)(B1 - 1)}, F3 = {3, 3};
    u32 t2[16], E1[2];
#pragma unroll
    for (int w = 0; w < 2; ++w) {
        u32 y[2];
#pragma unroll
        for (int a = 0; a < 2; ++a) {
            u32 x = 0;
#pragma unroll
            for (int i = 0; i < 4; ++i) {
                const int k = w * 8 + a * 4 + i;
                const u16x2 t = B2 - __builtin_bit_cast(u16x2, (d[k] >> 7) & 0x00ff00ffu);
                t2[k] = __builtin_bit_cast(u32, t);
                x |= __builtin_bit_cast(u32, __builtin_elementwise_min(t, F3)) << (4 * i);
            }
            y[a] = x | (x >> 14);
        }
        E1[w] = perm(y[1], y[0], 0x05040100u);
    }
    const u32 cnt = __builtin_popcount(E1[0] & (E1[0] >> 1) & 0x55555555u) + __builtin_popcount(E1[1] & (E1[1] >> 1) & 0x55555555u);
    const u32 inc = wave_scan(cnt);
    const u32 n = (u32)__builtin_amdgcn_readlane((int)inc, 63);
    bool raw = n > 1024u;
    const u32 tl = scr + 1152;
    if (!raw) {
        u32 q = scr + inc - cnt;
#define ESC2(T) "v_cmpx_lt_u16_e32 vcc, 2, " T "\n\tds_write_b8 %0, " T "\n\tv_add_u32 %0, 1, %0\n\ts_mov_b64 exec, -1\n\t" \
                "v_cmpx_lt_u32_e32 vcc, %17, " T "\n\tds_write_b8_d16_hi %0, " T "\n\tv_add_u32 %0, 1, %0\n\ts_mov_b64 exec, -1\n\t"
        asm volatile(ESC2("%1") ESC2("%2") ESC2("%3") ESC2("%4") ESC2("%5") ESC2("%6") ESC2("%7") ESC2("%8")
                     ESC2("%9") ESC2("%10") ESC2("%11") ESC2("%12") ESC2("%13") ESC2("%14") ESC2("%15") ESC2("%16") "s_nop 4"
                     : "+v"(q) : "v"(t2[0]), "v"(t2[1]), "v"(t2[2]), "v"(t2[3]), "v"(t2[4]), "v"(t2[5]), "v"(t2[6]), "v"(t2[7]),
                       "v"(t2[8]), "v"(t2[9]), "v"(t2[10]), "v"(t2[11]), "v"(t2[12]), "v"(t2[13]), "v"(t2[14]), "v"(t2[15]), "s"(0x2ffffu) : "vcc", "memory");
        if (lane < 16) *(LAS unsigned char*)(scr + n + lane) = 3;
    }
    asm volatile("" ::: "memory");
    const u32x4 ub = *(LAS u32x4*)(scr + lane * 16);
    asm volatile("" ::: "memory");
    u32x4 sm[2];
#pragma unroll
    for (int j = 0; j < 8; ++j) {
        const u32 H = perm(d[2 * j + 1], d[2 * j], 0x07050301u), L = perm(d[2 * j + 1], d[2 * j], 0x06040200u);
        sm[j >> 2][j & 3] = bfi(0x80808080u, H, L);
    }
    asm volatile("global_store_dwordx4 %0, %2, off sc0 sc1\n\tglobal_store_dwordx4 %0, %3, off offset:1024 sc0 sc1\n\t"
                 "global_store_dwordx2 %1, %4, off offset:2048 sc0 sc1\n\ts_nop 1"
                 :: "v"(pk16), "v"(pk8), "v"(sm[0]), "v"(sm[1]), "v"(u32x2{E1[0], E1[1]}) : "memory");
    u32 z[4], bad[4];
#pragma unroll
    for (int k = 0; k < 4; ++k) {
        const u32 u7 = (ub[k] & 0x7f7f7f7fu) + 0x01010101u;
        z[k] = ((u7 & 0x07070707u) + ((u7 >> 3) & 0x01010101u)) & 0x07070707u;
        bad[k] = (((u7 & 0x7f7f7f7fu) + 0x75757575u) | (u7 ^ ub[k])) & 0x80808080u;
    }
    u32 pm = (bad[0] >> 7) | (bad[1] >> 6) | (bad[2] >> 5) | (bad[3] >> 4);
    if (lane * 16 >= n) pm = 0;
    u32 np = 0;
    for (u64 mk = __ballot(pm != 0); mk != 0 && np <= NPATCH; mk = __ballot(pm != 0)) {
        const u32 bit = __builtin_ctz(pm | 0x80000000u), e = lane * 16 + (bit & 3) * 4 + (bit >> 3);
        const u32 ex = B1 - 1 - *(LAS unsigned char*)(scr + e);
        const u32 slot = np + __builtin_amdgcn_mbcnt_hi((u32)(mk >> 32), __builtin_amdgcn_mbcnt_lo((u32)mk, 0u));
        const u32 pa = tl + ((pm != 0 && slot < NPATCH) ? 252 - slot * 4 : 512 + lane * 4), pv = e | ((ex & 254u) << 15) | (ex << 23);
        *(LAS u32*)pa = pv; *(LAS u32*)(pa + 256) = pv;
        np += __builtin_popcountll(mk);
        pm &= pm - 1;
    }
    asm volatile("" ::: "memory");
    if (lane * 16 < n) {
        const u32 nib = z[3] | ((z[2] & 0x04040404u) << 1);
        const u32 ca = tl + 4 + (lane >> 1) * 12;
        *(LAS u32*)(ca + (lane & 1) * 4) = z[0] | (z[1] << 3) | (z[2] << 6 & 0xc0c0c0c0u);
        *(LAS unsigned short*)(ca + 8 + (lane & 1) * 2) = (unsigned short)perm(0u, nib | (nib >> 4), 0x0c0c0200u);
    }
    const u32 kind = (raw || np > NPATCH) ? 0x80000000u : (4 + 12 * ((n + 31) >> 5) + 4 * np > 256) ? 0x40000000u : 0u;
    if (lane == 0) *(LAS u32*)tl = B1 | ((np & 63u) << 8) | kind;
    asm volatile("" ::: "memory");
    if (lane < 32) asm volatile("global_store_dwordx4 %0, %1, off offset:2560 sc0 sc1\n\ts_nop 1" :: "v"(pk16), "v"(*(LAS u32x4*)(tl + lane * 16)) : "memory");
}

struct Packed { u32x4 sm0, sm1; u32x2 E1; u32 cA, cB, hp; };
__device__ __forceinline__ void read_packed(Packed& k, u32 a16, u32 a8, u32 ac, u32 ad, u32 a4, u32 stage) {
    asm volatile("ds_read_b64 %2, %7 offset:2048\n\tds_read_b32 %5, %10 offset:2560\n\tds_read_b32 %3, %8 offset:2564\n\tds_read_u16 %4, %9 offset:2572\n\t"
                 "ds_read_b128 %0, %6\n\tds_read_b128 %1, %6 offset:1024\n\ts_waitcnt lgkmcnt(0)"
                 : "=&v"(k.sm0), "=&v"(k.sm1), "=&v"(k.E1), "=&v"(k.cA), "=&v"(k.cB), "=&v"(k.hp)
                 : "v"(stage + a16), "v"(stage + a8), "v"(stage + ac), "v"(stage + ad), "v"(stage + a4) : "memory");
}
__device__ __forceinline__ void decode_block(const Packed& k, u32 pmask, u32 T1, u32 T2l, u32 T2h, u32 lane, u32 strm, char* dst, char* pflag, u32 call) {
    u32 sel[8];
#pragma unroll
    for (int j = 0; j < 8; ++j) sel[j] = *(LAS u32*)(L_LUT + ((k.E1[j >> 2] >> (8 * (j & 3))) & 255u) * 4);
    const u32 t = perm(0u, k.cB, 0x01010000u), u = bfi(0x00ff00ffu, t, t >> 4);
    *(LAS u32x4*)(strm + lane * 16) = u32x4{perm(T2h, T2l, k.cA & 0x07070707u), perm(T2h, T2l, (k.cA >> 3) & 0x07070707u),
                                            perm(T2h, T2l, bfi(0x03030303u, k.cA >> 6, u >> 1) & 0x07070707u), perm(T2h, T2l, u & 0x07070707u)};
    asm volatile("s_mov_b32 exec_lo, 0\n\ts_mov_b32 exec_hi, %2\n\tds_write_b8_d16_hi %0, %1\n\ts_mov_b64 exec, -1" :: "v"(strm + (k.hp & 0xffffu)), "v"(k.hp), "s"(pmask) : "memory");
    const u32 M0 = k.E1[0] & (k.E1[0] >> 1) & 0x55555555u, M1 = k.E1[1] & (k.E1[1] >> 1) & 0x55555555u;
    const u32 c0 = __builtin_popcount(M0), cnt = c0 + __builtin_popcount(M1);
    const u32 base0 = strm + wave_scan(cnt) - cnt, base1 = base0 + c0;
    u32 ad[8], w0[8], w1[8];
#pragma unroll
    for (int j = 0; j < 8; ++j) {
        const u32 M = j < 4 ? M0 : M1, bs = j < 4 ? base0 : base1;
        ad[j] = (j & 3) ? bs + __builtin_popcount(M & ((1u << (8 * (j & 3))) - 1u)) : bs;
    }
#pragma unroll
    for (int j = 0; j < 8; ++j) { w0[j] = *(LAS u32*)(ad[j] & ~3u); w1[j] = *(LAS u32*)((ad[j] & ~3u) + 4); }
    __builtin_amdgcn_sched_barrier(0);
    if (pflag) { publish(pflag, call, lane); }
#pragma unroll
    for (int f = 0; f < 4; ++f) {
        u32x4 o;
#pragma unroll
        for (int j = 2 * f; j < 2 * f + 2; ++j) {
            const u32 R = perm(__builtin_amdgcn_alignbyte(w1[j], w0[j], ad[j]), T1, sel[j]);
            const u32 S = j < 4 ? k.sm0[j & 3] : k.sm1[j & 3];
            const u32 H = bfi(0x80808080u, S, R), L = bfi(0x80808080u, R, S);
            o[(j & 1) * 2] = perm(H, L, 0x05010400u);
            o[(j & 1) * 2 + 1] = perm(H, L, 0x07030602u);
        }
        asm volatile("global_store_dwordx4 %0, %1, off offset:%c2 sc1\n\ts_nop 1" :: "v"(dst), "v"(o), "n"(f * 1024) : "memory");
    }
}
__device__ __forceinline__ u32 quad_of(u32 i) { return ((i >> 5) - G0) * 8 + (i & 7); }
__device__ __forceinline__ u32 lut_entry(u32 t) {
    u32 s = 0, ne = 4;
#pragma unroll
    for (int i = 0; i < 4; ++i) { const u32 c = (t >> (2 * i)) & 3u; s |= (c < 3 ? c + 1 : ne) << (8 * i); ne += c < 3 ? 0u : 1u; }
    return s;
}

#define POLL_THROTTLE 7
#define WBURST 72
#define TAILMASK "0xffff"
#define PF_END (16 * NGEMM_PER - NQUAD)
#define SPEC_AT 128
#define WFROM 18
#define PF_PRE 4
#define PF_DELAY 8
#define PACE 285
#define PACE0 1000
#define PRE_SPIN 8
#define INFL 3
#define EARLY_WG 32
#define POLL_ROWS 3
#define POLL_SLEEP 2
#define AMAX_SLEEP 1
#define KLOOP_ASM \
    "s_mov_b32 %[sbad], 0\n" \
    "s_mov_b32 %[sacc], 0\n" \
    "v_xor_b32 v85, 8, v84\n" \
    "v_mov_b64 v[96:97], 0\n" \
    "v_mov_b64 v[98:99], 0\n" \
    "v_mov_b64 v[100:101], 0\n" \
    "v_mov_b64 v[102:103], 0\n" \
    "v_mov_b64 v[104:105], 0\n" \
    "v_mov_b64 v[106:107], 0\n" \
    "v_mov_b64 v[108:109], 0\n" \
    "v_mov_b64 v[110:111], 0\n" \
    "v_mov_b64 v[112:113], 0\n" \
    "v_mov_b64 v[114:115], 0\n" \
    "v_mov_b64 v[116:117], 0\n" \
    "v_mov_b64 v[118:119], 0\n" \
    "v_mov_b64 v[120:121], 0\n" \
    "v_mov_b64 v[122:123], 0\n" \
    "v_mov_b64 v[124:125], 0\n" \
    "v_mov_b64 v[126:127], 0\n" \
    "v_mov_b64 v[128:129], 0\n" \
    "v_mov_b64 v[130:131], 0\n" \
    "v_mov_b64 v[132:133], 0\n" \
    "v_mov_b64 v[134:135], 0\n" \
    "v_mov_b64 v[136:137], 0\n" \
    "v_mov_b64 v[138:139], 0\n" \
    "v_mov_b64 v[140:141], 0\n" \
    "v_mov_b64 v[142:143], 0\n" \
    "v_mov_b64 v[144:145], 0\n" \
    "v_mov_b64 v[146:147], 0\n" \
    "v_mov_b64 v[148:149], 0\n" \
    "v_mov_b64 v[150:151], 0\n" \
    "v_mov_b64 v[152:153], 0\n" \
    "v_mov_b64 v[154:155], 0\n" \
    "v_mov_b64 v[156:157], 0\n" \
    "v_mov_b64 v[158:159], 0\n" \
    "v_mov_b64 v[160:161], 0\n" \
    "v_mov_b64 v[162:163], 0\n" \
    "v_mov_b64 v[164:165], 0\n" \
    "v_mov_b64 v[166:167], 0\n" \
    "v_mov_b64 v[168:169], 0\n" \
    "v_mov_b64 v[170:171], 0\n" \
    "v_mov_b64 v[172:173], 0\n" \
    "v_mov_b64 v[174:175], 0\n" \
    "v_mov_b64 v[176:177], 0\n" \
    "v_mov_b64 v[178:179], 0\n" \
    "v_mov_b64 v[180:181], 0\n" \
    "v_mov_b64 v[182:183], 0\n" \
    "v_mov_b64 v[184:185], 0\n" \
    "v_mov_b64 v[186:187], 0\n" \
    "v_mov_b64 v[188:189], 0\n" \
    "v_mov_b64 v[190:191], 0\n" \
    "v_mov_b64 v[192:193], 0\n" \
    "v_mov_b64 v[194:195], 0\n" \
    "v_mov_b64 v[196:197], 0\n" \
    "v_mov_b64 v[198:199], 0\n" \
    "v_mov_b64 v[200:201], 0\n" \
    "v_mov_b64 v[202:203], 0\n" \
    "v_mov_b64 v[204:205], 0\n" \
    "v_mov_b64 v[206:207], 0\n" \
    "v_mov_b64 v[208:209], 0\n" \
    "v_mov_b64 v[210:211], 0\n" \
    "v_mov_b64 v[212:213], 0\n" \
    "v_mov_b64 v[214:215], 0\n" \
    "v_mov_b64 v[216:217], 0\n" \
    "v_mov_b64 v[218:219], 0\n" \
    "v_mov_b64 v[220:221], 0\n" \
    "v_mov_b64 v[222:223], 0\n" \
    "v_mov_b64 v[224:225], 0\n" \
    "v_mov_b64 v[226:227], 0\n" \
    "v_mov_b64 v[228:229], 0\n" \
    "v_mov_b64 v[230:231], 0\n" \
    "v_mov_b64 v[232:233], 0\n" \
    "v_mov_b64 v[234:235], 0\n" \
    "v_mov_b64 v[236:237], 0\n" \
    "v_mov_b64 v[238:239], 0\n" \
    "v_mov_b64 v[240:241], 0\n" \
    "v_mov_b64 v[242:243], 0\n" \
    "v_mov_b64 v[244:245], 0\n" \
    "v_mov_b64 v[246:247], 0\n" \
    "v_mov_b64 v[248:249], 0\n" \
    "v_mov_b64 v[250:251], 0\n" \
    "v_mov_b64 v[252:253], 0\n" \
    "v_mov_b64 v[254:255], 0\n" \
    "s_mov_b32 %[sn], 3\n" \
    "s_branch Lp0_slow%=\n" \
    "Lp0_back%=:\n" \
    "s_mov_b32 %[sx], 0\n" \
    "buffer_load_dwordx4 v[0:3], %[xoff], %[xrs], %[sx] offen offset:0 sc0 sc1\n" \
    "buffer_load_dwordx4 v[4:7], %[xoff], %[xrs], %[sx] offen offset:1024 sc0 sc1\n" \
    "buffer_load_dwordx4 v[8:11], %[xoff], %[xrs], %[sx] offen offset:2048 sc0 sc1\n" \
    "buffer_load_dwordx4 v[12:15], %[xoff], %[xrs], %[sx] offen offset:3072 sc0 sc1\n" \
    "buffer_load_dwordx4 v[16:19], %[xoff4], %[xrs], %[sx] offen sc0 sc1\n" \
    "s_mov_b32 %[sx], 266240\n" \
    "buffer_load_dwordx4 v[20:23], %[xoff], %[xrs], %[sx] offen offset:0 sc0 sc1\n" \
    "buffer_load_dwordx4 v[24:27], %[xoff], %[xrs], %[sx] offen offset:1024 sc0 sc1\n" \
    "buffer_load_dwordx4 v[28:31], %[xoff], %[xrs], %[sx] offen offset:2048 sc0 sc1\n" \
    "buffer_load_dwordx4 v[32:35], %[xoff], %[xrs], %[sx] offen offset:3072 sc0 sc1\n" \
    "buffer_load_dwordx4 v[36:39], %[xoff4], %[xrs], %[sx] offen sc0 sc1\n" \
    "s_mov_b32 %[sx], 532480\n" \
    "buffer_load_dwordx4 v[40:43], %[xoff], %[xrs], %[sx] offen offset:0 sc0 sc1\n" \
    "buffer_load_dwordx4 v[44:47], %[xoff], %[xrs], %[sx] offen offset:1024 sc0 sc1\n" \
    "buffer_load_dwordx4 v[48:51], %[xoff], %[xrs], %[sx] offen offset:2048 sc0 sc1\n" \
    "buffer_load_dwordx4 v[52:55], %[xoff], %[xrs], %[sx] offen offset:3072 sc0 sc1\n" \
    "buffer_load_dwordx4 v[56:59], %[xoff4], %[xrs], %[sx] offen sc0 sc1\n" \
    "s_branch Lp1_slow%=\n" \
    "Lp1_back%=:\n" \
    "ds_read_b64_tr_b16 v[60:61], v84 offset:0\n" \
    "ds_read_b64_tr_b16 v[62:63], v84 offset:256\n" \
    "ds_read_b64_tr_b16 v[64:65], v85 offset:0\n" \
    "ds_read_b64_tr_b16 v[66:67], v85 offset:256\n" \
    "ds_read_b64_tr_b16 v[68:69], v84 offset:512\n" \
    "ds_read_b64_tr_b16 v[70:71], v84 offset:768\n" \
    "ds_read_b64_tr_b16 v[72:73], v85 offset:512\n" \
    "ds_read_b64_tr_b16 v[74:75], v85 offset:768\n" \
    "s_mov_b32 %[sn], 4\n" \
    "s_mov_b32 %[sring], 5\n" \
    "s_branch Lentry%=\n" \
    ".p2align 6\n" \
    "Lloop%=:\n" \
    "v_mfma_f32_16x16x32_bf16 v[152:155], v[60:63], v[56:59], v[152:155]\n" \
    "v_mfma_f32_16x16x32_bf16 v[172:175], v[64:67], v[56:59], v[172:175]\n" \
    "s_waitcnt lgkmcnt(5)\n" \
    "v_mfma_f32_16x16x32_bf16 v[176:179], v[68:71], v[40:43], v[176:179]\n" \
    "ds_read_b64_tr_b16 v[60:61], v84 offset:0\n" \
    "v_mfma_f32_16x16x32_bf16 v[196:199], v[72:75], v[40:43], v[196:199]\n" \
    "v_mfma_f32_16x16x32_bf16 v[200:203], v[72:75], v[44:47], v[200:203]\n" \
    "ds_read_b64_tr_b16 v[62:63], v84 offset:256\n" \
    "v_mfma_f32_16x16x32_bf16 v[180:183], v[68:71], v[44:47], v[180:183]\n" \
    "v_mfma_f32_16x16x32_bf16 v[184:187], v[68:71], v[48:51], v[184:187]\n" \
    "ds_read_b64_tr_b16 v[64:65], v85 offset:0\n" \
    "v_mfma_f32_16x16x32_bf16 v[204:207], v[72:75], v[48:51], v[204:207]\n" \
    "v_mfma_f32_16x16x32_bf16 v[208:211], v[72:75], v[52:55], v[208:211]\n" \
    "ds_read_b64_tr_b16 v[66:67], v85 offset:256\n" \
    "v_mfma_f32_16x16x32_bf16 v[188:191], v[68:71], v[52:55], v[188:191]\n" \
    "s_add_u32 %[sx], %[sx], %[xst]\n" \
    "v_mfma_f32_16x16x32_bf16 v[192:195], v[68:71], v[56:59], v[192:195]\n" \
    "v_mfma_f32_16x16x32_bf16 v[212:215], v[72:75], v[56:59], v[212:215]\n" \
    "s_waitcnt lgkmcnt(5)\n" \
    "v_mfma_f32_16x16x32_bf16 v[216:219], v[76:79], v[40:43], v[216:219]\n" \
    "ds_read_b64_tr_b16 v[68:69], v84 offset:512\n" \
    "v_mfma_f32_16x16x32_bf16 v[236:239], v[80:83], v[40:43], v[236:239]\n" \
    "buffer_load_dwordx4 v[40:43], %[xoff], %[xrs], %[sx] offen offset:0 sc0 sc1\n" \
    "v_mfma_f32_16x16x32_bf16 v[240:243], v[80:83], v[44:47], v[240:243]\n" \
    "ds_read_b64_tr_b16 v[70:71], v84 offset:768\n" \
    "v_mfma_f32_16x16x32_bf16 v[220:223], v[76:79], v[44:47], v[220:223]\n" \
    "buffer_load_dwordx4 v[44:47], %[xoff], %[xrs], %[sx] offen offset:1024 sc0 sc1\n" \
    "v_mfma_f32_16x16x32_bf16 v[224:227], v[76:79], v[48:51], v[224:227]\n" \
    "ds_read_b64_tr_b16 v[72:73], v85 offset:512\n" \
    "v_mfma_f32_16x16x32_bf16 v[244:247], v[80:83], v[48:51], v[244:247]\n" \
    "buffer_load_dwordx4 v[48:51], %[xoff], %[xrs], %[sx] offen offset:2048 sc0 sc1\n" \
    "v_mfma_f32_16x16x32_bf16 v[248:251], v[80:83], v[52:55], v[248:251]\n" \
    "ds_read_b64_tr_b16 v[74:75], v85 offset:768\n" \
    "v_mfma_f32_16x16x32_bf16 v[228:231], v[76:79], v[52:55], v[228:231]\n" \
    "buffer_load_dwordx4 v[52:55], %[xoff], %[xrs], %[sx] offen offset:3072 sc0 sc1\n" \
    "v_mfma_f32_16x16x32_bf16 v[232:235], v[76:79], v[56:59], v[232:235]\n" \
    "s_add_u32 %[sn], %[sn], 1\n" \
    "v_mfma_f32_16x16x32_bf16 v[252:255], v[80:83], v[56:59], v[252:255]\n" \
    "buffer_load_dwordx4 v[56:59], %[xoff4], %[xrs], %[sx] offen sc0 sc1\n" \
    "Lentry%=:\n" \
    "s_waitcnt vmcnt(14) lgkmcnt(4)\n" \
    "v_mfma_f32_16x16x32_bf16 v[96:99], v[60:63], v[0:3], v[96:99]\n" \
    "ds_read_b64_tr_b16 v[76:77], v84 offset:4096\n" \
    "v_mfma_f32_16x16x32_bf16 v[116:119], v[64:67], v[0:3], v[116:119]\n" \
    "s_waitcnt vmcnt(13)\n" \
    "v_mfma_f32_16x16x32_bf16 v[120:123], v[64:67], v[4:7], v[120:123]\n" \
    "ds_read_b64_tr_b16 v[78:79], v84 offset:4352\n" \
    "v_mfma_f32_16x16x32_bf16 v[100:103], v[60:63], v[4:7], v[100:103]\n" \
    "s_waitcnt vmcnt(12)\n" \
    "v_mfma_f32_16x16x32_bf16 v[104:107], v[60:63], v[8:11], v[104:107]\n" \
    "ds_read_b64_tr_b16 v[80:81], v85 offset:4096\n" \
    "v_mfma_f32_16x16x32_bf16 v[124:127], v[64:67], v[8:11], v[124:127]\n" \
    "s_waitcnt vmcnt(11)\n" \
    "v_mfma_f32_16x16x32_bf16 v[128:131], v[64:67], v[12:15], v[128:131]\n" \
    "ds_read_b64_tr_b16 v[82:83], v85 offset:4352\n" \
    "v_mfma_f32_16x16x32_bf16 v[108:111], v[60:63], v[12:15], v[108:111]\n" \
    "s_waitcnt vmcnt(10)\n" \
    "v_mfma_f32_16x16x32_bf16 v[112:115], v[60:63], v[16:19], v[112:115]\n" \
    "ds_read_b64 v[88:89], %[actl]\n" \
    "v_mfma_f32_16x16x32_bf16 v[132:135], v[64:67], v[16:19], v[132:135]\n" \
    "s_waitcnt lgkmcnt(5)\n" \
    "v_mfma_f32_16x16x32_bf16 v[136:139], v[68:71], v[0:3], v[136:139]\n" \
    "ds_read_b64_tr_b16 v[60:61], v84 offset:4608\n" \
    "v_mfma_f32_16x16x32_bf16 v[156:159], v[72:75], v[0:3], v[156:159]\n" \
    "v_mfma_f32_16x16x32_bf16 v[160:163], v[72:75], v[4:7], v[160:163]\n" \
    "ds_read_b64_tr_b16 v[62:63], v84 offset:4864\n" \
    "v_mfma_f32_16x16x32_bf16 v[140:143], v[68:71], v[4:7], v[140:143]\n" \
    "v_mfma_f32_16x16x32_bf16 v[144:147], v[68:71], v[8:11], v[144:147]\n" \
    "ds_read_b64_tr_b16 v[64:65], v85 offset:4608\n" \
    "v_mfma_f32_16x16x32_bf16 v[164:167], v[72:75], v[8:11], v[164:167]\n" \
    "v_mfma_f32_16x16x32_bf16 v[168:171], v[72:75], v[12:15], v[168:171]\n" \
    "ds_read_b64_tr_b16 v[66:67], v85 offset:4864\n" \
    "v_mfma_f32_16x16x32_bf16 v[148:151], v[68:71], v[12:15], v[148:151]\n" \
    "s_add_u32 %[sx], %[sx], %[xst]\n" \
    "v_mfma_f32_16x16x32_bf16 v[152:155], v[68:71], v[16:19], v[152:155]\n" \
    "v_mfma_f32_16x16x32_bf16 v[172:175], v[72:75], v[16:19], v[172:175]\n" \
    "s_waitcnt lgkmcnt(4)\n" \
    "v_mfma_f32_16x16x32_bf16 v[176:179], v[76:79], v[0:3], v[176:179]\n" \
    "ds_read_b64_tr_b16 v[68:69], v84 offset:8192\n" \
    "v_mfma_f32_16x16x32_bf16 v[196:199], v[80:83], v[0:3], v[196:199]\n" \
    "v_readfirstlane_b32 %[sxr], v88\n" \
    "v_mfma_f32_16x16x32_bf16 v[200:203], v[80:83], v[4:7], v[200:203]\n" \
    "ds_read_b64_tr_b16 v[70:71], v84 offset:8448\n" \
    "v_mfma_f32_16x16x32_bf16 v[180:183], v[76:79], v[4:7], v[180:183]\n" \
    "v_mfma_f32_16x16x32_bf16 v[184:187], v[76:79], v[8:11], v[184:187]\n" \
    "ds_read_b64_tr_b16 v[72:73], v85 offset:8192\n" \
    "s_cmp_ge_u32 %[sxr], %[sn]\n" \
    "v_mfma_f32_16x16x32_bf16 v[204:207], v[80:83], v[8:11], v[204:207]\n" \
    "s_cbranch_scc0 Lx00_slow%=\n" \
    "Lx00_back%=:\n" \
    "v_mfma_f32_16x16x32_bf16 v[208:211], v[80:83], v[12:15], v[208:211]\n" \
    "ds_read_b64_tr_b16 v[74:75], v85 offset:8448\n" \
    "v_mfma_f32_16x16x32_bf16 v[188:191], v[76:79], v[12:15], v[188:191]\n" \
    "v_mfma_f32_16x16x32_bf16 v[192:195], v[76:79], v[16:19], v[192:195]\n" \
    "v_mfma_f32_16x16x32_bf16 v[212:215], v[80:83], v[16:19], v[212:215]\n" \
    "s_waitcnt lgkmcnt(4)\n" \
    "v_mfma_f32_16x16x32_bf16 v[216:219], v[60:63], v[0:3], v[216:219]\n" \
    "ds_read_b64_tr_b16 v[76:77], v84 offset:8704\n" \
    "v_mfma_f32_16x16x32_bf16 v[236:239], v[64:67], v[0:3], v[236:239]\n" \
    "buffer_load_dwordx4 v[0:3], %[xoff], %[xrs], %[sx] offen offset:0 sc0 sc1\n" \
    "v_mfma_f32_16x16x32_bf16 v[240:243], v[64:67], v[4:7], v[240:243]\n" \
    "ds_read_b64_tr_b16 v[78:79], v84 offset:8960\n" \
    "v_mfma_f32_16x16x32_bf16 v[220:223], v[60:63], v[4:7], v[220:223]\n" \
    "buffer_load_dwordx4 v[4:7], %[xoff], %[xrs], %[sx] offen offset:1024 sc0 sc1\n" \
    "v_mfma_f32_16x16x32_bf16 v[224:227], v[60:63], v[8:11], v[224:227]\n" \
    "ds_read_b64_tr_b16 v[80:81], v85 offset:8704\n" \
    "v_mfma_f32_16x16x32_bf16 v[244:247], v[64:67], v[8:11], v[244:247]\n" \
    "buffer_load_dwordx4 v[8:11], %[xoff], %[xrs], %[sx] offen offset:2048 sc0 sc1\n" \
    "v_mfma_f32_16x16x32_bf16 v[248:251], v[64:67], v[12:15], v[248:251]\n" \
    "ds_read_b64_tr_b16 v[82:83], v85 offset:8960\n" \
    "v_mfma_f32_16x16x32_bf16 v[228:231], v[60:63], v[12:15], v[228:231]\n" \
    "buffer_load_dwordx4 v[12:15], %[xoff], %[xrs], %[sx] offen offset:3072 sc0 sc1\n" \
    "v_mfma_f32_16x16x32_bf16 v[232:235], v[60:63], v[16:19], v[232:235]\n" \
    "s_add_u32 %[sn], %[sn], 1\n" \
    "v_mfma_f32_16x16x32_bf16 v[252:255], v[64:67], v[16:19], v[252:255]\n" \
    "buffer_load_dwordx4 v[16:19], %[xoff4], %[xrs], %[sx] offen sc0 sc1\n" \
    "s_waitcnt vmcnt(14) lgkmcnt(4)\n" \
    "v_mfma_f32_16x16x32_bf16 v[96:99], v[68:71], v[20:23], v[96:99]\n" \
    "ds_read_b64_tr_b16 v[60:61], v84 offset:12288\n" \
    "v_mfma_f32_16x16x32_bf16 v[116:119], v[72:75], v[20:23], v[116:119]\n" \
    "s_waitcnt vmcnt(13)\n" \
    "v_mfma_f32_16x16x32_bf16 v[120:123], v[72:75], v[24:27], v[120:123]\n" \
    "ds_read_b64_tr_b16 v[62:63], v84 offset:12544\n" \
    "v_mfma_f32_16x16x32_bf16 v[100:103], v[68:71], v[24:27], v[100:103]\n" \
    "s_waitcnt vmcnt(12)\n" \
    "v_mfma_f32_16x16x32_bf16 v[104:107], v[68:71], v[28:31], v[104:107]\n" \
    "ds_read_b64_tr_b16 v[64:65], v85 offset:12288\n" \
    "v_mfma_f32_16x16x32_bf16 v[124:127], v[72:75], v[28:31], v[124:127]\n" \
    "s_waitcnt vmcnt(11)\n" \
    "v_mfma_f32_16x16x32_bf16 v[128:131], v[72:75], v[32:35], v[128:131]\n" \
    "ds_read_b64_tr_b16 v[66:67], v85 offset:12544\n" \
    "v_mfma_f32_16x16x32_bf16 v[108:111], v[68:71], v[32:35], v[108:111]\n" \
    "s_waitcnt vmcnt(10)\n" \
    "v_mfma_f32_16x16x32_bf16 v[112:115], v[68:71], v[36:39], v[112:115]\n" \
    "ds_read_b64 v[88:89], %[actl]\n" \
    "v_mfma_f32_16x16x32_bf16 v[132:135], v[72:75], v[36:39], v[132:135]\n" \
    "s_waitcnt lgkmcnt(5)\n" \
    "v_mfma_f32_16x16x32_bf16 v[136:139], v[76:79], v[20:23], v[136:139]\n" \
    "ds_read_b64_tr_b16 v[68:69], v84 offset:12800\n" \
    "v_mfma_f32_16x16x32_bf16 v[156:159], v[80:83], v[20:23], v[156:159]\n" \
    "s_sub_u32 %[sring], %[sring], 1\n" \
    "v_mfma_f32_16x16x32_bf16 v[160:163], v[80:83], v[24:27], v[160:163]\n" \
    "ds_read_b64_tr_b16 v[70:71], v84 offset:13056\n" \
    "v_mfma_f32_16x16x32_bf16 v[140:143], v[76:79], v[24:27], v[140:143]\n" \
    "s_cselect_b32 %[stmp], %[dwrap], %[dstep]\n" \
    "v_mfma_f32_16x16x32_bf16 v[144:147], v[76:79], v[28:31], v[144:147]\n" \
    "ds_read_b64_tr_b16 v[72:73], v85 offset:12800\n" \
    "v_mfma_f32_16x16x32_bf16 v[164:167], v[80:83], v[28:31], v[164:167]\n" \
    "s_cselect_b32 %[sring], 5, %[sring]\n" \
    "v_mfma_f32_16x16x32_bf16 v[168:171], v[80:83], v[32:35], v[168:171]\n" \
    "ds_read_b64_tr_b16 v[74:75], v85 offset:13056\n" \
    "v_mfma_f32_16x16x32_bf16 v[148:151], v[76:79], v[32:35], v[148:151]\n" \
    "s_add_u32 %[sx], %[sx], %[xst]\n" \
    "v_mfma_f32_16x16x32_bf16 v[152:155], v[76:79], v[36:39], v[152:155]\n" \
    "v_mfma_f32_16x16x32_bf16 v[172:175], v[80:83], v[36:39], v[172:175]\n" \
    "s_waitcnt lgkmcnt(4)\n" \
    "v_mfma_f32_16x16x32_bf16 v[176:179], v[60:63], v[20:23], v[176:179]\n" \
    "ds_read_b64_tr_b16 v[76:77], v84 offset:16384\n" \
    "v_mfma_f32_16x16x32_bf16 v[196:199], v[64:67], v[20:23], v[196:199]\n" \
    "v_readfirstlane_b32 %[sxr], v88\n" \
    "v_mfma_f32_16x16x32_bf16 v[200:203], v[64:67], v[24:27], v[200:203]\n" \
    "ds_read_b64_tr_b16 v[78:79], v84 offset:16640\n" \
    "v_mfma_f32_16x16x32_bf16 v[180:183], v[60:63], v[24:27], v[180:183]\n" \
    "v_mfma_f32_16x16x32_bf16 v[184:187], v[60:63], v[28:31], v[184:187]\n" \
    "ds_read_b64_tr_b16 v[80:81], v85 offset:16384\n" \
    "v_mfma_f32_16x16x32_bf16 v[204:207], v[64:67], v[28:31], v[204:207]\n" \
    "s_cmp_ge_u32 %[sxr], %[sn]\n" \
    "v_mfma_f32_16x16x32_bf16 v[208:211], v[64:67], v[32:35], v[208:211]\n" \
    "ds_read_b64_tr_b16 v[82:83], v85 offset:16640\n" \
    "s_cbranch_scc0 Lx10_slow%=\n" \
    "Lx10_back%=:\n" \
    "v_mfma_f32_16x16x32_bf16 v[188:191], v[60:63], v[32:35], v[188:191]\n" \
    "v_mfma_f32_16x16x32_bf16 v[192:195], v[60:63], v[36:39], v[192:195]\n" \
    "v_mfma_f32_16x16x32_bf16 v[212:215], v[64:67], v[36:39], v[212:215]\n" \
    "s_waitcnt lgkmcnt(4)\n" \
    "v_mfma_f32_16x16x32_bf16 v[216:219], v[68:71], v[20:23], v[216:219]\n" \
    "ds_read_b64_tr_b16 v[60:61], v84 offset:16896\n" \
    "v_mfma_f32_16x16x32_bf16 v[236:239], v[72:75], v[20:23], v[236:239]\n" \
    "buffer_load_dwordx4 v[20:23], %[xoff], %[xrs], %[sx] offen offset:0 sc0 sc1\n" \
    "v_mfma_f32_16x16x32_bf16 v[240:243], v[72:75], v[24:27], v[240:243]\n" \
    "ds_read_b64_tr_b16 v[62:63], v84 offset:17152\n" \
    "v_mfma_f32_16x16x32_bf16 v[220:223], v[68:71], v[24:27], v[220:223]\n" \
    "buffer_load_dwordx4 v[24:27], %[xoff], %[xrs], %[sx] offen offset:1024 sc0 sc1\n" \
    "v_mfma_f32_16x16x32_bf16 v[224:227], v[68:71], v[28:31], v[224:227]\n" \
    "ds_read_b64_tr_b16 v[64:65], v85 offset:16896\n" \
    "v_mfma_f32_16x16x32_bf16 v[244:247], v[72:75], v[28:31], v[244:247]\n" \
    "buffer_load_dwordx4 v[28:31], %[xoff], %[xrs], %[sx] offen offset:2048 sc0 sc1\n" \
    "v_mfma_f32_16x16x32_bf16 v[248:251], v[72:75], v[32:35], v[248:251]\n" \
    "ds_read_b64_tr_b16 v[66:67], v85 offset:17152\n" \
    "v_mfma_f32_16x16x32_bf16 v[228:231], v[68:71], v[32:35], v[228:231]\n" \
    "buffer_load_dwordx4 v[32:35], %[xoff], %[xrs], %[sx] offen offset:3072 sc0 sc1\n" \
    "v_mfma_f32_16x16x32_bf16 v[232:235], v[68:71], v[36:39], v[232:235]\n" \
    "ds_read_b64 v[88:89], %[actl]\n" \
    "s_add_u32 %[sn], %[sn], 1\n" \
    "v_mfma_f32_16x16x32_bf16 v[252:255], v[72:75], v[36:39], v[252:255]\n" \
    "buffer_load_dwordx4 v[36:39], %[xoff4], %[xrs], %[sx] offen sc0 sc1\n" \
    "s_waitcnt vmcnt(14) lgkmcnt(5)\n" \
    "v_mfma_f32_16x16x32_bf16 v[96:99], v[76:79], v[40:43], v[96:99]\n" \
    "ds_read_b64_tr_b16 v[68:69], v84 offset:20480\n" \
    "v_mfma_f32_16x16x32_bf16 v[116:119], v[80:83], v[40:43], v[116:119]\n" \
    "s_waitcnt vmcnt(13)\n" \
    "v_mfma_f32_16x16x32_bf16 v[120:123], v[80:83], v[44:47], v[120:123]\n" \
    "ds_read_b64_tr_b16 v[70:71], v84 offset:20736\n" \
    "v_mfma_f32_16x16x32_bf16 v[100:103], v[76:79], v[44:47], v[100:103]\n" \
    "s_waitcnt vmcnt(12)\n" \
    "v_mfma_f32_16x16x32_bf16 v[104:107], v[76:79], v[48:51], v[104:107]\n" \
    "ds_read_b64_tr_b16 v[72:73], v85 offset:20480\n" \
    "v_mfma_f32_16x16x32_bf16 v[124:127], v[80:83], v[48:51], v[124:127]\n" \
    "s_waitcnt vmcnt(11)\n" \
    "v_mfma_f32_16x16x32_bf16 v[128:131], v[80:83], v[52:55], v[128:131]\n" \
    "ds_read_b64_tr_b16 v[74:75], v85 offset:20736\n" \
    "v_mfma_f32_16x16x32_bf16 v[108:111], v[76:79], v[52:55], v[108:111]\n" \
    "s_waitcnt vmcnt(10)\n" \
    "v_mfma_f32_16x16x32_bf16 v[112:115], v[76:79], v[56:59], v[112:115]\n" \
    "v_mov_b32 v92, %[sn]\n" \
    "v_mfma_f32_16x16x32_bf16 v[132:135], v[80:83], v[56:59], v[132:135]\n" \
    "s_waitcnt lgkmcnt(4)\n" \
    "v_mfma_f32_16x16x32_bf16 v[136:139], v[60:63], v[40:43], v[136:139]\n" \
    "ds_read_b64_tr_b16 v[76:77], v84 offset:20992\n" \
    "v_min_u32 v88, v88, v89\n" \
    "v_mfma_f32_16x16x32_bf16 v[156:159], v[64:67], v[40:43], v[156:159]\n" \
    "ds_read_b64_tr_b16 v[78:79], v84 offset:21248\n" \
    "v_mfma_f32_16x16x32_bf16 v[160:163], v[64:67], v[44:47], v[160:163]\n" \
    "ds_read_b64_tr_b16 v[80:81], v85 offset:20992\n" \
    "v_readfirstlane_b32 %[sxr], v88\n" \
    "v_mfma_f32_16x16x32_bf16 v[140:143], v[60:63], v[44:47], v[140:143]\n" \
    "ds_read_b64_tr_b16 v[82:83], v85 offset:21248\n" \
    "v_mfma_f32_16x16x32_bf16 v[144:147], v[60:63], v[48:51], v[144:147]\n" \
    "ds_write_b32 %[acn], v92\n" \
    "v_mfma_f32_16x16x32_bf16 v[164:167], v[64:67], v[48:51], v[164:167]\n" \
    "v_add_u32 v84, %[stmp], v84\n" \
    "s_cmp_ge_u32 %[sxr], %[sn]\n" \
    "v_mfma_f32_16x16x32_bf16 v[168:171], v[64:67], v[52:55], v[168:171]\n" \
    "v_add_u32 v85, %[stmp], v85\n" \
    "s_cbranch_scc0 Lx20_slow%=\n" \
    "Lx20_back%=:\n" \
    "v_mfma_f32_16x16x32_bf16 v[148:151], v[60:63], v[52:55], v[148:151]\n" \
    "v_mfma_f32_16x16x32_bf16 v[152:155], v[60:63], v[56:59], v[152:155]\n" \
    "v_mfma_f32_16x16x32_bf16 v[172:175], v[64:67], v[56:59], v[172:175]\n" \
    "s_waitcnt lgkmcnt(5)\n" \
    "v_mfma_f32_16x16x32_bf16 v[176:179], v[68:71], v[40:43], v[176:179]\n" \
    "ds_read_b64_tr_b16 v[60:61], v84 offset:0\n" \
    "v_mfma_f32_16x16x32_bf16 v[196:199], v[72:75], v[40:43], v[196:199]\n" \
    "v_mfma_f32_16x16x32_bf16 v[200:203], v[72:75], v[44:47], v[200:203]\n" \
    "ds_read_b64_tr_b16 v[62:63], v84 offset:256\n" \
    "v_mfma_f32_16x16x32_bf16 v[180:183], v[68:71], v[44:47], v[180:183]\n" \
    "v_mfma_f32_16x16x32_bf16 v[184:187], v[68:71], v[48:51], v[184:187]\n" \
    "ds_read_b64_tr_b16 v[64:65], v85 offset:0\n" \
    "v_mfma_f32_16x16x32_bf16 v[204:207], v[72:75], v[48:51], v[204:207]\n" \
    "v_mfma_f32_16x16x32_bf16 v[208:211], v[72:75], v[52:55], v[208:211]\n" \
    "ds_read_b64_tr_b16 v[66:67], v85 offset:256\n" \
    "v_mfma_f32_16x16x32_bf16 v[188:191], v[68:71], v[52:55], v[188:191]\n" \
    "s_add_u32 %[sx], %[sx], %[xst]\n" \
    "v_mfma_f32_16x16x32_bf16 v[192:195], v[68:71], v[56:59], v[192:195]\n" \
    "v_mfma_f32_16x16x32_bf16 v[212:215], v[72:75], v[56:59], v[212:215]\n" \
    "s_waitcnt lgkmcnt(5)\n" \
    "v_mfma_f32_16x16x32_bf16 v[216:219], v[76:79], v[40:43], v[216:219]\n" \
    "ds_read_b64_tr_b16 v[68:69], v84 offset:512\n" \
    "v_mfma_f32_16x16x32_bf16 v[236:239], v[80:83], v[40:43], v[236:239]\n" \
    "buffer_load_dwordx4 v[40:43], %[xoff], %[xrs], %[sx] offen offset:0 sc0 sc1\n" \
    "v_mfma_f32_16x16x32_bf16 v[240:243], v[80:83], v[44:47], v[240:243]\n" \
    "ds_read_b64_tr_b16 v[70:71], v84 offset:768\n" \
    "v_mfma_f32_16x16x32_bf16 v[220:223], v[76:79], v[44:47], v[220:223]\n" \
    "buffer_load_dwordx4 v[44:47], %[xoff], %[xrs], %[sx] offen offset:1024 sc0 sc1\n" \
    "v_mfma_f32_16x16x32_bf16 v[224:227], v[76:79], v[48:51], v[224:227]\n" \
    "ds_read_b64_tr_b16 v[72:73], v85 offset:512\n" \
    "v_mfma_f32_16x16x32_bf16 v[244:247], v[80:83], v[48:51], v[244:247]\n" \
    "buffer_load_dwordx4 v[48:51], %[xoff], %[xrs], %[sx] offen offset:2048 sc0 sc1\n" \
    "v_mfma_f32_16x16x32_bf16 v[248:251], v[80:83], v[52:55], v[248:251]\n" \
    "ds_read_b64_tr_b16 v[74:75], v85 offset:768\n" \
    "v_mfma_f32_16x16x32_bf16 v[228:231], v[76:79], v[52:55], v[228:231]\n" \
    "buffer_load_dwordx4 v[52:55], %[xoff], %[xrs], %[sx] offen offset:3072 sc0 sc1\n" \
    "v_mfma_f32_16x16x32_bf16 v[232:235], v[76:79], v[56:59], v[232:235]\n" \
    "s_add_u32 %[sn], %[sn], 1\n" \
    "v_mfma_f32_16x16x32_bf16 v[252:255], v[80:83], v[56:59], v[252:255]\n" \
    "buffer_load_dwordx4 v[56:59], %[xoff4], %[xrs], %[sx] offen sc0 sc1\n" \
    "s_waitcnt vmcnt(14) lgkmcnt(4)\n" \
    "v_mfma_f32_16x16x32_bf16 v[96:99], v[60:63], v[0:3], v[96:99]\n" \
    "ds_read_b64_tr_b16 v[76:77], v84 offset:4096\n" \
    "v_mfma_f32_16x16x32_bf16 v[116:119], v[64:67], v[0:3], v[116:119]\n" \
    "s_waitcnt vmcnt(13)\n" \
    "v_mfma_f32_16x16x32_bf16 v[120:123], v[64:67], v[4:7], v[120:123]\n" \
    "ds_read_b64_tr_b16 v[78:79], v84 offset:4352\n" \
    "v_mfma_f32_16x16x32_bf16 v[100:103], v[60:63], v[4:7], v[100:103]\n" \
    "s_waitcnt vmcnt(12)\n" \
    "v_mfma_f32_16x16x32_bf16 v[104:107], v[60:63], v[8:11], v[104:107]\n" \
    "ds_read_b64_tr_b16 v[80:81], v85 offset:4096\n" \
    "v_mfma_f32_16x16x32_bf16 v[124:127], v[64:67], v[8:11], v[124:127]\n" \
    "s_waitcnt vmcnt(11)\n" \
    "v_mfma_f32_16x16x32_bf16 v[128:131], v[64:67], v[12:15], v[128:131]\n" \
    "ds_read_b64_tr_b16 v[82:83], v85 offset:4352\n" \
    "v_mfma_f32_16x16x32_bf16 v[108:111], v[60:63], v[12:15], v[108:111]\n" \
    "s_waitcnt vmcnt(10)\n" \
    "v_mfma_f32_16x16x32_bf16 v[112:115], v[60:63], v[16:19], v[112:115]\n" \
    "ds_read_b64 v[88:89], %[actl]\n" \
    "v_mfma_f32_16x16x32_bf16 v[132:135], v[64:67], v[16:19], v[132:135]\n" \
    "s_waitcnt lgkmcnt(5)\n" \
    "v_mfma_f32_16x16x32_bf16 v[136:139], v[68:71], v[0:3], v[136:139]\n" \
    "ds_read_b64_tr_b16 v[60:61], v84 offset:4608\n" \
    "v_mfma_f32_16x16x32_bf16 v[156:159], v[72:75], v[0:3], v[156:159]\n" \
    "v_mfma_f32_16x16x32_bf16 v[160:163], v[72:75], v[4:7], v[160:163]\n" \
    "ds_read_b64_tr_b16 v[62:63], v84 offset:4864\n" \
    "v_mfma_f32_16x16x32_bf16 v[140:143], v[68:71], v[4:7], v[140:143]\n" \
    "v_mfma_f32_16x16x32_bf16 v[144:147], v[68:71], v[8:11], v[144:147]\n" \
    "ds_read_b64_tr_b16 v[64:65], v85 offset:4608\n" \
    "v_mfma_f32_16x16x32_bf16 v[164:167], v[72:75], v[8:11], v[164:167]\n" \
    "v_mfma_f32_16x16x32_bf16 v[168:171], v[72:75], v[12:15], v[168:171]\n" \
    "ds_read_b64_tr_b16 v[66:67], v85 offset:4864\n" \
    "v_mfma_f32_16x16x32_bf16 v[148:151], v[68:71], v[12:15], v[148:151]\n" \
    "s_add_u32 %[sx], %[sx], %[xst]\n" \
    "v_mfma_f32_16x16x32_bf16 v[152:155], v[68:71], v[16:19], v[152:155]\n" \
    "v_mfma_f32_16x16x32_bf16 v[172:175], v[72:75], v[16:19], v[172:175]\n" \
    "s_waitcnt lgkmcnt(4)\n" \
    "v_mfma_f32_16x16x32_bf16 v[176:179], v[76:79], v[0:3], v[176:179]\n" \
    "ds_read_b64_tr_b16 v[68:69], v84 offset:8192\n" \
    "v_mfma_f32_16x16x32_bf16 v[196:199], v[80:83], v[0:3], v[196:199]\n" \
    "v_readfirstlane_b32 %[sxr], v88\n" \
    "v_mfma_f32_16x16x32_bf16 v[200:203], v[80:83], v[4:7], v[200:203]\n" \
    "ds_read_b64_tr_b16 v[70:71], v84 offset:8448\n" \
    "v_mfma_f32_16x16x32_bf16 v[180:183], v[76:79], v[4:7], v[180:183]\n" \
    "v_mfma_f32_16x16x32_bf16 v[184:187], v[76:79], v[8:11], v[184:187]\n" \
    "ds_read_b64_tr_b16 v[72:73], v85 offset:8192\n" \
    "v_mfma_f32_16x16x32_bf16 v[204:207], v[80:83], v[8:11], v[204:207]\n" \
    "s_cmp_ge_u32 %[sxr], %[sn]\n" \
    "v_mfma_f32_16x16x32_bf16 v[208:211], v[80:83], v[12:15], v[208:211]\n" \
    "ds_read_b64_tr_b16 v[74:75], v85 offset:8448\n" \
    "s_cbranch_scc0 Lx01_slow%=\n" \
    "Lx01_back%=:\n" \
    "v_mfma_f32_16x16x32_bf16 v[188:191], v[76:79], v[12:15], v[188:191]\n" \
    "v_mfma_f32_16x16x32_bf16 v[192:195], v[76:79], v[16:19], v[192:195]\n" \
    "v_mfma_f32_16x16x32_bf16 v[212:215], v[80:83], v[16:19], v[212:215]\n" \
    "s_waitcnt lgkmcnt(4)\n" \
    "v_mfma_f32_16x16x32_bf16 v[216:219], v[60:63], v[0:3], v[216:219]\n" \
    "ds_read_b64_tr_b16 v[76:77], v84 offset:8704\n" \
    "v_mfma_f32_16x16x32_bf16 v[236:239], v[64:67], v[0:3], v[236:239]\n" \
    "buffer_load_dwordx4 v[0:3], %[xoff], %[xrs], %[sx] offen offset:0 sc0 sc1\n" \
    "v_mfma_f32_16x16x32_bf16 v[240:243], v[64:67], v[4:7], v[240:243]\n" \
    "ds_read_b64_tr_b16 v[78:79], v84 offset:8960\n" \
    "v_mfma_f32_16x16x32_bf16 v[220:223], v[60:63], v[4:7], v[220:223]\n" \
    "buffer_load_dwordx4 v[4:7], %[xoff], %[xrs], %[sx] offen offset:1024 sc0 sc1\n" \
    "v_mfma_f32_16x16x32_bf16 v[224:227], v[60:63], v[8:11], v[224:227]\n" \
    "ds_read_b64_tr_b16 v[80:81], v85 offset:8704\n" \
    "v_mfma_f32_16x16x32_bf16 v[244:247], v[64:67], v[8:11], v[244:247]\n" \
    "buffer_load_dwordx4 v[8:11], %[xoff], %[xrs], %[sx] offen offset:2048 sc0 sc1\n" \
    "v_mfma_f32_16x16x32_bf16 v[248:251], v[64:67], v[12:15], v[248:251]\n" \
    "ds_read_b64_tr_b16 v[82:83], v85 offset:8960\n" \
    "v_mfma_f32_16x16x32_bf16 v[228:231], v[60:63], v[12:15], v[228:231]\n" \
    "buffer_load_dwordx4 v[12:15], %[xoff], %[xrs], %[sx] offen offset:3072 sc0 sc1\n" \
    "v_mfma_f32_16x16x32_bf16 v[232:235], v[60:63], v[16:19], v[232:235]\n" \
    "s_add_u32 %[sn], %[sn], 1\n" \
    "v_mfma_f32_16x16x32_bf16 v[252:255], v[64:67], v[16:19], v[252:255]\n" \
    "buffer_load_dwordx4 v[16:19], %[xoff4], %[xrs], %[sx] offen sc0 sc1\n" \
    "s_waitcnt vmcnt(14) lgkmcnt(4)\n" \
    "v_mfma_f32_16x16x32_bf16 v[96:99], v[68:71], v[20:23], v[96:99]\n" \
    "ds_read_b64_tr_b16 v[60:61], v84 offset:12288\n" \
    "v_mfma_f32_16x16x32_bf16 v[116:119], v[72:75], v[20:23], v[116:119]\n" \
    "s_waitcnt vmcnt(13)\n" \
    "v_mfma_f32_16x16x32_bf16 v[120:123], v[72:75], v[24:27], v[120:123]\n" \
    "ds_read_b64_tr_b16 v[62:63], v84 offset:12544\n" \
    "v_mfma_f32_16x16x32_bf16 v[100:103], v[68:71], v[24:27], v[100:103]\n" \
    "s_waitcnt vmcnt(12)\n" \
    "v_mfma_f32_16x16x32_bf16 v[104:107], v[68:71], v[28:31], v[104:107]\n" \
    "ds_read_b64_tr_b16 v[64:65], v85 offset:12288\n" \
    "v_mfma_f32_16x16x32_bf16 v[124:127], v[72:75], v[28:31], v[124:127]\n" \
    "s_waitcnt vmcnt(11)\n" \
    "v_mfma_f32_16x16x32_bf16 v[128:131], v[72:75], v[32:35], v[128:131]\n" \
    "ds_read_b64_tr_b16 v[66:67], v85 offset:12544\n" \
    "v_mfma_f32_16x16x32_bf16 v[108:111], v[68:71], v[32:35], v[108:111]\n" \
    "s_waitcnt vmcnt(10)\n" \
    "v_mfma_f32_16x16x32_bf16 v[112:115], v[68:71], v[36:39], v[112:115]\n" \
    "ds_read_b64 v[88:89], %[actl]\n" \
    "v_mfma_f32_16x16x32_bf16 v[132:135], v[72:75], v[36:39], v[132:135]\n" \
    "s_waitcnt lgkmcnt(5)\n" \
    "v_mfma_f32_16x16x32_bf16 v[136:139], v[76:79], v[20:23], v[136:139]\n" \
    "ds_read_b64_tr_b16 v[68:69], v84 offset:12800\n" \
    "v_mfma_f32_16x16x32_bf16 v[156:159], v[80:83], v[20:23], v[156:159]\n" \
    "s_sub_u32 %[sring], %[sring], 1\n" \
    "v_mfma_f32_16x16x32_bf16 v[160:163], v[80:83], v[24:27], v[160:163]\n" \
    "ds_read_b64_tr_b16 v[70:71], v84 offset:13056\n" \
    "v_mfma_f32_16x16x32_bf16 v[140:143], v[76:79], v[24:27], v[140:143]\n" \
    "s_cselect_b32 %[stmp], %[dwrap], %[dstep]\n" \
    "v_mfma_f32_16x16x32_bf16 v[144:147], v[76:79], v[28:31], v[144:147]\n" \
    "ds_read_b64_tr_b16 v[72:73], v85 offset:12800\n" \
    "v_mfma_f32_16x16x32_bf16 v[164:167], v[80:83], v[28:31], v[164:167]\n" \
    "s_cselect_b32 %[sring], 5, %[sring]\n" \
    "v_mfma_f32_16x16x32_bf16 v[168:171], v[80:83], v[32:35], v[168:171]\n" \
    "ds_read_b64_tr_b16 v[74:75], v85 offset:13056\n" \
    "v_mfma_f32_16x16x32_bf16 v[148:151], v[76:79], v[32:35], v[148:151]\n" \
    "s_add_u32 %[sx], %[sx], %[xst]\n" \
    "v_mfma_f32_16x16x32_bf16 v[152:155], v[76:79], v[36:39], v[152:155]\n" \
    "v_mfma_f32_16x16x32_bf16 v[172:175], v[80:83], v[36:39], v[172:175]\n" \
    "s_waitcnt lgkmcnt(4)\n" \
    "v_mfma_f32_16x16x32_bf16 v[176:179], v[60:63], v[20:23], v[176:179]\n" \
    "ds_read_b64_tr_b16 v[76:77], v84 offset:16384\n" \
    "v_mfma_f32_16x16x32_bf16 v[196:199], v[64:67], v[20:23], v[196:199]\n" \
    "v_readfirstlane_b32 %[sxr], v88\n" \
    "v_mfma_f32_16x16x32_bf16 v[200:203], v[64:67], v[24:27], v[200:203]\n" \
    "ds_read_b64_tr_b16 v[78:79], v84 offset:16640\n" \
    "v_mfma_f32_16x16x32_bf16 v[180:183], v[60:63], v[24:27], v[180:183]\n" \
    "v_mfma_f32_16x16x32_bf16 v[184:187], v[60:63], v[28:31], v[184:187]\n" \
    "ds_read_b64_tr_b16 v[80:81], v85 offset:16384\n" \
    "s_cmp_ge_u32 %[sxr], %[sn]\n" \
    "v_mfma_f32_16x16x32_bf16 v[204:207], v[64:67], v[28:31], v[204:207]\n" \
    "s_cbranch_scc0 Lx11_slow%=\n" \
    "Lx11_back%=:\n" \
    "v_mfma_f32_16x16x32_bf16 v[208:211], v[64:67], v[32:35], v[208:211]\n" \
    "ds_read_b64_tr_b16 v[82:83], v85 offset:16640\n" \
    "v_mfma_f32_16x16x32_bf16 v[188:191], v[60:63], v[32:35], v[188:191]\n" \
    "v_mfma_f32_16x16x32_bf16 v[192:195], v[60:63], v[36:39], v[192:195]\n" \
    "v_mfma_f32_16x16x32_bf16 v[212:215], v[64:67], v[36:39], v[212:215]\n" \
    "s_waitcnt lgkmcnt(4)\n" \
    "v_mfma_f32_16x16x32_bf16 v[216:219], v[68:71], v[20:23], v[216:219]\n" \
    "ds_read_b64_tr_b16 v[60:61], v84 offset:16896\n" \
    "v_mfma_f32_16x16x32_bf16 v[236:239], v[72:75], v[20:23], v[236:239]\n" \
    "buffer_load_dwordx4 v[20:23], %[xoff], %[xrs], %[sx] offen offset:0 sc0 sc1\n" \
    "v_mfma_f32_16x16x32_bf16 v[240:243], v[72:75], v[24:27], v[240:243]\n" \
    "ds_read_b64_tr_b16 v[62:63], v84 offset:17152\n" \
    "v_mfma_f32_16x16x32_bf16 v[220:223], v[68:71], v[24:27], v[220:223]\n" \
    "buffer_load_dwordx4 v[24:27], %[xoff], %[xrs], %[sx] offen offset:1024 sc0 sc1\n" \
    "v_mfma_f32_16x16x32_bf16 v[224:227], v[68:71], v[28:31], v[224:227]\n" \
    "ds_read_b64_tr_b16 v[64:65], v85 offset:16896\n" \
    "v_mfma_f32_16x16x32_bf16 v[244:247], v[72:75], v[28:31], v[244:247]\n" \
    "buffer_load_dwordx4 v[28:31], %[xoff], %[xrs], %[sx] offen offset:2048 sc0 sc1\n" \
    "v_mfma_f32_16x16x32_bf16 v[248:251], v[72:75], v[32:35], v[248:251]\n" \
    "ds_read_b64_tr_b16 v[66:67], v85 offset:17152\n" \
    "v_mfma_f32_16x16x32_bf16 v[228:231], v[68:71], v[32:35], v[228:231]\n" \
    "buffer_load_dwordx4 v[32:35], %[xoff], %[xrs], %[sx] offen offset:3072 sc0 sc1\n" \
    "v_mfma_f32_16x16x32_bf16 v[232:235], v[68:71], v[36:39], v[232:235]\n" \
    "ds_read_b64 v[88:89], %[actl]\n" \
    "s_add_u32 %[sn], %[sn], 1\n" \
    "v_mfma_f32_16x16x32_bf16 v[252:255], v[72:75], v[36:39], v[252:255]\n" \
    "buffer_load_dwordx4 v[36:39], %[xoff4], %[xrs], %[sx] offen sc0 sc1\n" \
    "s_waitcnt vmcnt(14) lgkmcnt(5)\n" \
    "v_mfma_f32_16x16x32_bf16 v[96:99], v[76:79], v[40:43], v[96:99]\n" \
    "ds_read_b64_tr_b16 v[68:69], v84 offset:20480\n" \
    "v_mfma_f32_16x16x32_bf16 v[116:119], v[80:83], v[40:43], v[116:119]\n" \
    "s_waitcnt vmcnt(13)\n" \
    "v_mfma_f32_16x16x32_bf16 v[120:123], v[80:83], v[44:47], v[120:123]\n" \
    "ds_read_b64_tr_b16 v[70:71], v84 offset:20736\n" \
    "v_mfma_f32_16x16x32_bf16 v[100:103], v[76:79], v[44:47], v[100:103]\n" \
    "s_waitcnt vmcnt(12)\n" \
    "v_mfma_f32_16x16x32_bf16 v[104:107], v[76:79], v[48:51], v[104:107]\n" \
    "ds_read_b64_tr_b16 v[72:73], v85 offset:20480\n" \
    "v_mfma_f32_16x16x32_bf16 v[124:127], v[80:83], v[48:51], v[124:127]\n" \
    "s_waitcnt vmcnt(11)\n" \
    "v_mfma_f32_16x16x32_bf16 v[128:131], v[80:83], v[52:55], v[128:131]\n" \
    "ds_read_b64_tr_b16 v[74:75], v85 offset:20736\n" \
    "v_mfma_f32_16x16x32_bf16 v[108:111], v[76:79], v[52:55], v[108:111]\n" \
    "s_waitcnt vmcnt(10)\n" \
    "v_mfma_f32_16x16x32_bf16 v[112:115], v[76:79], v[56:59], v[112:115]\n" \
    "v_mov_b32 v92, %[sn]\n" \
    "v_mfma_f32_16x16x32_bf16 v[132:135], v[80:83], v[56:59], v[132:135]\n" \
    "s_waitcnt lgkmcnt(4)\n" \
    "v_mfma_f32_16x16x32_bf16 v[136:139], v[60:63], v[40:43], v[136:139]\n" \
    "ds_read_b64_tr_b16 v[76:77], v84 offset:20992\n" \
    "v_min_u32 v88, v88, v89\n" \
    "v_mfma_f32_16x16x32_bf16 v[156:159], v[64:67], v[40:43], v[156:159]\n" \
    "ds_read_b64_tr_b16 v[78:79], v84 offset:21248\n" \
    "v_mfma_f32_16x16x32_bf16 v[160:163], v[64:67], v[44:47], v[160:163]\n" \
    "ds_read_b64_tr_b16 v[80:81], v85 offset:20992\n" \
    "v_readfirstlane_b32 %[sxr], v88\n" \
    "v_mfma_f32_16x16x32_bf16 v[140:143], v[60:63], v[44:47], v[140:143]\n" \
    "ds_read_b64_tr_b16 v[82:83], v85 offset:21248\n" \
    "v_mfma_f32_16x16x32_bf16 v[144:147], v[60:63], v[48:51], v[144:147]\n" \
    "ds_write_b32 %[acn], v92\n" \
    "v_mfma_f32_16x16x32_bf16 v[164:167], v[64:67], v[48:51], v[164:167]\n" \
    "v_add_u32 v84, %[stmp], v84\n" \
    "v_mfma_f32_16x16x32_bf16 v[168:171], v[64:67], v[52:55], v[168:171]\n" \
    "v_add_u32 v85, %[stmp], v85\n" \
    "s_cmp_ge_u32 %[sxr], %[sn]\n" \
    "v_mfma_f32_16x16x32_bf16 v[148:151], v[60:63], v[52:55], v[148:151]\n" \
    "s_cbranch_scc1 Lloop%=\n" \
    "Lx21_slow%=:\n" \
    "s_lshl_b32 %[scnt], %[sbad], 24\n" \
    "Lx21_loop%=:\n" \
    "ds_read_b64 v[88:89], %[actl]\n" \
    "s_waitcnt lgkmcnt(0)\n" \
    "v_min_u32 v88, v88, v89\n" \
    "s_add_u32 %[scnt], %[scnt], 1\n" \
    "v_readfirstlane_b32 %[sxr], v88\n" \
    "s_add_u32 %[sacc], %[sacc], 1\n" \
    "s_cmp_ge_u32 %[sxr], %[sn]\n" \
    "s_cbranch_scc1 Lloop%=\n" \
    "s_sleep 1\n" \
    "s_cmp_lt_u32 %[scnt], 0x1000000\n" \
    "s_cbranch_scc1 Lx21_loop%=\n" \
    "s_mov_b32 %[sbad], 1\n" \
    "s_branch Lloop%=\n" \
    "Lx00_slow%=:\n" \
    "s_cmpk_eq_u32 %[sn], 130\n" \
    "s_cbranch_scc1 Ltail%=\n" \
    "s_lshl_b32 %[scnt], %[sbad], 24\n" \
    "Lx00_loop%=:\n" \
    "ds_read_b64 v[88:89], %[actl]\n" \
    "s_waitcnt lgkmcnt(0)\n" \
    "s_add_u32 %[scnt], %[scnt], 1\n" \
    "v_readfirstlane_b32 %[sxr], v88\n" \
    "s_add_u32 %[sacc], %[sacc], 1\n" \
    "s_cmp_ge_u32 %[sxr], %[sn]\n" \
    "s_cbranch_scc1 Lx00_back%=\n" \
    "s_sleep 1\n" \
    "s_cmp_lt_u32 %[scnt], 0x1000000\n" \
    "s_cbranch_scc1 Lx00_loop%=\n" \
    "s_mov_b32 %[sbad], 1\n" \
    "s_branch Lx00_back%=\n" \
    "Lx10_slow%=:\n" \
    "s_lshl_b32 %[scnt], %[sbad], 24\n" \
    "Lx10_loop%=:\n" \
    "ds_read_b64 v[88:89], %[actl]\n" \
    "s_waitcnt lgkmcnt(0)\n" \
    "s_add_u32 %[scnt], %[scnt], 1\n" \
    "v_readfirstlane_b32 %[sxr], v88\n" \
    "s_add_u32 %[sacc], %[sacc], 1\n" \
    "s_cmp_ge_u32 %[sxr], %[sn]\n" \
    "s_cbranch_scc1 Lx10_back%=\n" \
    "s_sleep 1\n" \
    "s_cmp_lt_u32 %[scnt], 0x1000000\n" \
    "s_cbranch_scc1 Lx10_loop%=\n" \
    "s_mov_b32 %[sbad], 1\n" \
    "s_branch Lx10_back%=\n" \
    "Lx20_slow%=:\n" \
    "s_lshl_b32 %[scnt], %[sbad], 24\n" \
    "Lx20_loop%=:\n" \
    "ds_read_b64 v[88:89], %[actl]\n" \
    "s_waitcnt lgkmcnt(0)\n" \
    "v_min_u32 v88, v88, v89\n" \
    "s_add_u32 %[scnt], %[scnt], 1\n" \
    "v_readfirstlane_b32 %[sxr], v88\n" \
    "s_add_u32 %[sacc], %[sacc], 1\n" \
    "s_cmp_ge_u32 %[sxr], %[sn]\n" \
    "s_cbranch_scc1 Lx20_back%=\n" \
    "s_sleep 1\n" \
    "s_cmp_lt_u32 %[scnt], 0x1000000\n" \
    "s_cbranch_scc1 Lx20_loop%=\n" \
    "s_mov_b32 %[sbad], 1\n" \
    "s_branch Lx20_back%=\n" \
    "Lx01_slow%=:\n" \
    "s_lshl_b32 %[scnt], %[sbad], 24\n" \
    "Lx01_loop%=:\n" \
    "ds_read_b64 v[88:89], %[actl]\n" \
    "s_waitcnt lgkmcnt(0)\n" \
    "s_add_u32 %[scnt], %[scnt], 1\n" \
    "v_readfirstlane_b32 %[sxr], v88\n" \
    "s_add_u32 %[sacc], %[sacc], 1\n" \
    "s_cmp_ge_u32 %[sxr], %[sn]\n" \
    "s_cbranch_scc1 Lx01_back%=\n" \
    "s_sleep 1\n" \
    "s_cmp_lt_u32 %[scnt], 0x1000000\n" \
    "s_cbranch_scc1 Lx01_loop%=\n" \
    "s_mov_b32 %[sbad], 1\n" \
    "s_branch Lx01_back%=\n" \
    "Lx11_slow%=:\n" \
    "s_lshl_b32 %[scnt], %[sbad], 24\n" \
    "Lx11_loop%=:\n" \
    "ds_read_b64 v[88:89], %[actl]\n" \
    "s_waitcnt lgkmcnt(0)\n" \
    "s_add_u32 %[scnt], %[scnt], 1\n" \
    "v_readfirstlane_b32 %[sxr], v88\n" \
    "s_add_u32 %[sacc], %[sacc], 1\n" \
    "s_cmp_ge_u32 %[sxr], %[sn]\n" \
    "s_cbranch_scc1 Lx11_back%=\n" \
    "s_sleep 1\n" \
    "s_cmp_lt_u32 %[scnt], 0x1000000\n" \
    "s_cbranch_scc1 Lx11_loop%=\n" \
    "s_mov_b32 %[sbad], 1\n" \
    "s_branch Lx11_back%=\n" \
    "Lp0_slow%=:\n" \
    "s_lshl_b32 %[scnt], %[sbad], 24\n" \
    "Lp0_loop%=:\n" \
    "ds_read_b64 v[88:89], %[actl]\n" \
    "s_waitcnt lgkmcnt(0)\n" \
    "s_add_u32 %[scnt], %[scnt], 1\n" \
    "v_readfirstlane_b32 %[sxr], v88\n" \
    "s_add_u32 %[sacc], %[sacc], 0\n" \
    "s_cmp_ge_u32 %[sxr], %[sn]\n" \
    "s_cbranch_scc1 Lp0_back%=\n" \
    "s_sleep 1\n" \
    "s_cmp_lt_u32 %[scnt], 0x1000000\n" \
    "s_cbranch_scc1 Lp0_loop%=\n" \
    "s_mov_b32 %[sbad], 1\n" \
    "s_branch Lp0_back%=\n" \
    "Lp1_slow%=:\n" \
    "s_lshl_b32 %[scnt], %[sbad], 24\n" \
    "Lp1_loop%=:\n" \
    "ds_read_b64 v[88:89], %[actl]\n" \
    "s_waitcnt lgkmcnt(0)\n" \
    "v_mov_b32 v88, v89\n" \
    "s_add_u32 %[scnt], %[scnt], 1\n" \
    "v_readfirstlane_b32 %[sxr], v88\n" \
    "s_add_u32 %[sacc], %[sacc], 0\n" \
    "s_cmp_ge_u32 %[sxr], %[sn]\n" \
    "s_cbranch_scc1 Lp1_back%=\n" \
    "s_sleep 1\n" \
    "s_cmp_lt_u32 %[scnt], 0x1000000\n" \
    "s_cbranch_scc1 Lp1_loop%=\n" \
    "s_mov_b32 %[sbad], 1\n" \
    "s_branch Lp1_back%=\n" \
    "Ltail%=:\n" \
    "v_mov_b32 v92, 0\n" \
    "v_mov_b32 v93, 0\n" \
    "v_mfma_f32_16x16x32_bf16 v[208:211], v[80:83], v[12:15], v[208:211]\n" \
    "ds_read_b64_tr_b16 v[74:75], v85 offset:8448\n" \
    "v_mfma_f32_16x16x32_bf16 v[188:191], v[76:79], v[12:15], v[188:191]\n" \
    "v_mfma_f32_16x16x32_bf16 v[192:195], v[76:79], v[16:19], v[192:195]\n" \
    "v_mfma_f32_16x16x32_bf16 v[212:215], v[80:83], v[16:19], v[212:215]\n" \
    "s_waitcnt lgkmcnt(4)\n" \
    "v_mfma_f32_16x16x32_bf16 v[216:219], v[60:63], v[0:3], v[216:219]\n" \
    "ds_read_b64_tr_b16 v[76:77], v84 offset:8704\n" \
    "v_mfma_f32_16x16x32_bf16 v[236:239], v[64:67], v[0:3], v[236:239]\n" \
    "v_mfma_f32_16x16x32_bf16 v[240:243], v[64:67], v[4:7], v[240:243]\n" \
    "ds_read_b64_tr_b16 v[78:79], v84 offset:8960\n" \
    "v_mfma_f32_16x16x32_bf16 v[220:223], v[60:63], v[4:7], v[220:223]\n" \
    "v_mfma_f32_16x16x32_bf16 v[224:227], v[60:63], v[8:11], v[224:227]\n" \
    "ds_read_b64_tr_b16 v[80:81], v85 offset:8704\n" \
    "v_mfma_f32_16x16x32_bf16 v[244:247], v[64:67], v[8:11], v[244:247]\n" \
    "v_mfma_f32_16x16x32_bf16 v[248:251], v[64:67], v[12:15], v[248:251]\n" \
    "ds_read_b64_tr_b16 v[82:83], v85 offset:8960\n" \
    "v_mfma_f32_16x16x32_bf16 v[228:231], v[60:63], v[12:15], v[228:231]\n" \
    "v_mfma_f32_16x16x32_bf16 v[232:235], v[60:63], v[16:19], v[232:235]\n" \
    "v_mfma_f32_16x16x32_bf16 v[252:255], v[64:67], v[16:19], v[252:255]\n" \
    "s_waitcnt vmcnt(9) lgkmcnt(4)\n" \
    "v_mfma_f32_16x16x32_bf16 v[96:99], v[68:71], v[20:23], v[96:99]\n" \
    "ds_read_b64_tr_b16 v[60:61], v84 offset:12288\n" \
    "v_mfma_f32_16x16x32_bf16 v[116:119], v[72:75], v[20:23], v[116:119]\n" \
    "s_waitcnt vmcnt(8)\n" \
    "v_mfma_f32_16x16x32_bf16 v[120:123], v[72:75], v[24:27], v[120:123]\n" \
    "ds_read_b64_tr_b16 v[62:63], v84 offset:12544\n" \
    "v_mfma_f32_16x16x32_bf16 v[100:103], v[68:71], v[24:27], v[100:103]\n" \
    "s_waitcnt vmcnt(7)\n" \
    "v_mfma_f32_16x16x32_bf16 v[104:107], v[68:71], v[28:31], v[104:107]\n" \
    "ds_read_b64_tr_b16 v[64:65], v85 offset:12288\n" \
    "v_mfma_f32_16x16x32_bf16 v[124:127], v[72:75], v[28:31], v[124:127]\n" \
    "s_waitcnt vmcnt(6)\n" \
    "v_mfma_f32_16x16x32_bf16 v[128:131], v[72:75], v[32:35], v[128:131]\n" \
    "ds_read_b64_tr_b16 v[66:67], v85 offset:12544\n" \
    "v_mfma_f32_16x16x32_bf16 v[108:111], v[68:71], v[32:35], v[108:111]\n" \
    "s_waitcnt vmcnt(5)\n" \
    "v_mfma_f32_16x16x32_bf16 v[112:115], v[68:71], v[36:39], v[112:115]\n" \
    "v_mfma_f32_16x16x32_bf16 v[132:135], v[72:75], v[36:39], v[132:135]\n" \
    "s_waitcnt lgkmcnt(4)\n" \
    "v_mfma_f32_16x16x32_bf16 v[136:139], v[76:79], v[20:23], v[136:139]\n" \
    "ds_read_b64_tr_b16 v[68:69], v84 offset:12800\n" \
    "v_max3_i32 v92, v92, v96, v97\n" \
    "v_max3_i32 v93, v93, v98, v99\n" \
    "v_mfma_f32_16x16x32_bf16 v[156:159], v[80:83], v[20:23], v[156:159]\n" \
    "v_max3_i32 v92, v92, v116, v117\n" \
    "v_max3_i32 v93, v93, v118, v119\n" \
    "v_mfma_f32_16x16x32_bf16 v[160:163], v[80:83], v[24:27], v[160:163]\n" \
    "ds_read_b64_tr_b16 v[70:71], v84 offset:13056\n" \
    "v_max3_i32 v92, v92, v120, v121\n" \
    "v_max3_i32 v93, v93, v122, v123\n" \
    "v_mfma_f32_16x16x32_bf16 v[140:143], v[76:79], v[24:27], v[140:143]\n" \
    "v_max3_i32 v92, v92, v100, v101\n" \
    "v_max3_i32 v93, v93, v102, v103\n" \
    "v_mfma_f32_16x16x32_bf16 v[144:147], v[76:79], v[28:31], v[144:147]\n" \
    "ds_read_b64_tr_b16 v[72:73], v85 offset:12800\n" \
    "v_max3_i32 v92, v92, v104, v105\n" \
    "v_max3_i32 v93, v93, v106, v107\n" \
    "v_mfma_f32_16x16x32_bf16 v[164:167], v[80:83], v[28:31], v[164:167]\n" \
    "v_max3_i32 v92, v92, v124, v125\n" \
    "v_max3_i32 v93, v93, v126, v127\n" \
    "v_mfma_f32_16x16x32_bf16 v[168:171], v[80:83], v[32:35], v[168:171]\n" \
    "ds_read_b64_tr_b16 v[74:75], v85 offset:13056\n" \
    "v_max3_i32 v92, v92, v128, v129\n" \
    "v_max3_i32 v93, v93, v130, v131\n" \
    "v_mfma_f32_16x16x32_bf16 v[148:151], v[76:79], v[32:35], v[148:151]\n" \
    "v_max3_i32 v92, v92, v108, v109\n" \
    "v_max3_i32 v93, v93, v110, v111\n" \
    "v_mfma_f32_16x16x32_bf16 v[152:155], v[76:79], v[36:39], v[152:155]\n" \
    "v_max3_i32 v92, v92, v112, v113\n" \
    "v_max3_i32 v93, v93, v114, v115\n" \
    "v_mfma_f32_16x16x32_bf16 v[172:175], v[80:83], v[36:39], v[172:175]\n" \
    "v_max3_i32 v92, v92, v132, v133\n" \
    "v_max3_i32 v93, v93, v134, v135\n" \
    "s_waitcnt lgkmcnt(4)\n" \
    "v_mfma_f32_16x16x32_bf16 v[176:179], v[60:63], v[20:23], v[176:179]\n" \
    "v_max3_i32 v92, v92, v136, v137\n" \
    "v_max3_i32 v93, v93, v138, v139\n" \
    "v_mfma_f32_16x16x32_bf16 v[196:199], v[64:67], v[20:23], v[196:199]\n" \
    "v_max3_i32 v92, v92, v156, v157\n" \
    "v_max3_i32 v93, v93, v158, v159\n" \
    "v_mfma_f32_16x16x32_bf16 v[200:203], v[64:67], v[24:27], v[200:203]\n" \
    "v_max3_i32 v92, v92, v160, v161\n" \
    "v_max3_i32 v93, v93, v162, v163\n" \
    "v_mfma_f32_16x16x32_bf16 v[180:183], v[60:63], v[24:27], v[180:183]\n" \
    "v_max3_i32 v92, v92, v140, v141\n" \
    "v_max3_i32 v93, v93, v142, v143\n" \
    "v_mfma_f32_16x16x32_bf16 v[184:187], v[60:63], v[28:31], v[184:187]\n" \
    "v_max3_i32 v92, v92, v144, v145\n" \
    "v_max3_i32 v93, v93, v146, v147\n" \
    "v_mfma_f32_16x16x32_bf16 v[204:207], v[64:67], v[28:31], v[204:207]\n" \
    "v_max3_i32 v92, v92, v164, v165\n" \
    "v_max3_i32 v93, v93, v166, v167\n" \
    "v_mfma_f32_16x16x32_bf16 v[208:211], v[64:67], v[32:35], v[208:211]\n" \
    "v_max3_i32 v92, v92, v168, v169\n" \
    "v_max3_i32 v93, v93, v170, v171\n" \
    "v_mfma_f32_16x16x32_bf16 v[188:191], v[60:63], v[32:35], v[188:191]\n" \
    "v_max3_i32 v92, v92, v148, v149\n" \
    "v_max3_i32 v93, v93, v150, v151\n" \
    "v_mfma_f32_16x16x32_bf16 v[192:195], v[60:63], v[36:39], v[192:195]\n" \
    "v_max3_i32 v92, v92, v152, v153\n" \
    "v_max3_i32 v93, v93, v154, v155\n" \
    "v_mfma_f32_16x16x32_bf16 v[212:215], v[64:67], v[36:39], v[212:215]\n" \
    "v_max3_i32 v92, v92, v172, v173\n" \
    "v_max3_i32 v93, v93, v174, v175\n" \
    "s_waitcnt lgkmcnt(0)\n" \
    "v_mfma_f32_16x16x32_bf16 v[216:219], v[68:71], v[20:23], v[216:219]\n" \
    "v_max3_i32 v92, v92, v176, v177\n" \
    "v_max3_i32 v93, v93, v178, v179\n" \
    "v_mfma_f32_16x16x32_bf16 v[236:239], v[72:75], v[20:23], v[236:239]\n" \
    "v_max3_i32 v92, v92, v196, v197\n" \
    "v_max3_i32 v93, v93, v198, v199\n" \
    "v_mfma_f32_16x16x32_bf16 v[240:243], v[72:75], v[24:27], v[240:243]\n" \
    "v_max3_i32 v92, v92, v200, v201\n" \
    "v_max3_i32 v93, v93, v202, v203\n" \
    "v_mfma_f32_16x16x32_bf16 v[220:223], v[68:71], v[24:27], v[220:223]\n" \
    "v_max3_i32 v92, v92, v180, v181\n" \
    "v_max3_i32 v93, v93, v182, v183\n" \
    "v_mfma_f32_16x16x32_bf16 v[224:227], v[68:71], v[28:31], v[224:227]\n" \
    "v_max3_i32 v92, v92, v184, v185\n" \
    "v_max3_i32 v93, v93, v186, v187\n" \
    "v_mfma_f32_16x16x32_bf16 v[244:247], v[72:75], v[28:31], v[244:247]\n" \
    "v_max3_i32 v92, v92, v204, v205\n" \
    "v_max3_i32 v93, v93, v206, v207\n" \
    "v_mfma_f32_16x16x32_bf16 v[248:251], v[72:75], v[32:35], v[248:251]\n" \
    "v_max3_i32 v92, v92, v208, v209\n" \
    "v_max3_i32 v93, v93, v210, v211\n" \
    "v_mfma_f32_16x16x32_bf16 v[228:231], v[68:71], v[32:35], v[228:231]\n" \
    "v_max3_i32 v92, v92, v188, v189\n" \
    "v_max3_i32 v93, v93, v190, v191\n" \
    "v_mfma_f32_16x16x32_bf16 v[232:235], v[68:71], v[36:39], v[232:235]\n" \
    "v_max3_i32 v92, v92, v192, v193\n" \
    "v_max3_i32 v93, v93, v194, v195\n" \
    "v_mfma_f32_16x16x32_bf16 v[252:255], v[72:75], v[36:39], v[252:255]\n" \
    "v_max3_i32 v92, v92, v212, v213\n" \
    "v_max3_i32 v93, v93, v214, v215\n" \
    "v_max3_i32 v92, v92, v216, v217\n" \
    "v_max3_i32 v93, v93, v218, v219\n" \
    "v_max3_i32 v92, v92, v236, v237\n" \
    "v_max3_i32 v93, v93, v238, v239\n" \
    "v_max3_i32 v92, v92, v240, v241\n" \
    "v_max3_i32 v93, v93, v242, v243\n" \
    "v_max3_i32 v92, v92, v220, v221\n" \
    "v_max3_i32 v93, v93, v222, v223\n" \
    "v_max3_i32 v92, v92, v224, v225\n" \
    "v_max3_i32 v93, v93, v226, v227\n" \
    "v_max3_i32 v92, v92, v244, v245\n" \
    "v_max3_i32 v93, v93, v246, v247\n" \
    "v_max3_i32 v92, v92, v248, v249\n" \
    "v_max3_i32 v93, v93, v250, v251\n" \
    "v_max3_i32 v92, v92, v228, v229\n" \
    "v_max3_i32 v93, v93, v230, v231\n" \
    "v_max3_i32 v92, v92, v232, v233\n" \
    "v_max3_i32 v93, v93, v234, v235\n" \
    "v_max3_i32 v92, v92, v252, v253\n" \
    "v_max3_i32 v93, v93, v254, v255\n" \
    "v_max_i32 v92, v92, v93\n" \
    "s_waitcnt vmcnt(0) lgkmcnt(0)\n" \
    ""
#define KLOOP_CLOBBERS "v0", "v1", "v2", "v3", "v4", "v5", "v6", "v7", "v8", "v9", "v10", "v11", "v12", "v13", "v14", "v15", "v16", "v17", "v18", "v19", "v20", "v21", "v22", "v23", "v24", "v25", "v26", "v27", "v28", "v29", "v30", "v31", "v32", "v33", "v34", "v35", "v36", "v37", "v38", "v39", "v40", "v41", "v42", "v43", "v44", "v45", "v46", "v47", "v48", "v49", "v50", "v51", "v52", "v53", "v54", "v55", "v56", "v57", "v58", "v59", "v60", "v61", "v62", "v63", "v64", "v65", "v66", "v67", "v68", "v69", "v70", "v71", "v72", "v73", "v74", "v75", "v76", "v77", "v78", "v79", "v80", "v81", "v82", "v83", "v85", "v88", "v89", "v93", "v94", "v95"
#define QUANT_ASM \
    "v_pk_mul_f32 v[0:1], %[inv], v[96:97]\n" \
    "v_pk_mul_f32 v[4:5], %[inv], v[116:117]\n" \
    "v_pk_mul_f32 v[8:9], %[inv], v[136:137]\n" \
    "v_pk_mul_f32 v[12:13], %[inv], v[156:157]\n" \
    "v_pk_mul_f32 v[2:3], %[inv], v[98:99]\n" \
    "v_pk_mul_f32 v[6:7], %[inv], v[118:119]\n" \
    "v_pk_mul_f32 v[10:11], %[inv], v[138:139]\n" \
    "v_pk_mul_f32 v[14:15], %[inv], v[158:159]\n" \
    "v_cvt_pk_u8_f32 v16, v0, 0, 0\n" \
    "v_cvt_pk_u8_f32 v17, v4, 0, 0\n" \
    "v_cvt_pk_u8_f32 v18, v8, 0, 0\n" \
    "v_cvt_pk_u8_f32 v19, v12, 0, 0\n" \
    "v_cvt_pk_u8_f32 v16, v1, 1, v16\n" \
    "v_cvt_pk_u8_f32 v17, v5, 1, v17\n" \
    "v_cvt_pk_u8_f32 v18, v9, 1, v18\n" \
    "v_cvt_pk_u8_f32 v19, v13, 1, v19\n" \
    "v_cvt_pk_u8_f32 v16, v2, 2, v16\n" \
    "v_cvt_pk_u8_f32 v17, v6, 2, v17\n" \
    "v_cvt_pk_u8_f32 v18, v10, 2, v18\n" \
    "v_cvt_pk_u8_f32 v19, v14, 2, v19\n" \
    "v_cvt_pk_u8_f32 v16, v3, 3, v16\n" \
    "v_cvt_pk_u8_f32 v17, v7, 3, v17\n" \
    "v_cvt_pk_u8_f32 v18, v11, 3, v18\n" \
    "v_cvt_pk_u8_f32 v19, v15, 3, v19\n" \
    "s_mov_b64 exec, %[odd]\n" \
    "v_swap_b32 v16, v17\n" \
    "v_swap_b32 v18, v19\n" \
    "s_mov_b64 exec, -1\n" \
    "global_store_dwordx4 %[o0], v[16:19], off offset:0 sc1\n" \
    "v_pk_mul_f32 v[20:21], %[inv], v[176:177]\n" \
    "v_pk_mul_f32 v[24:25], %[inv], v[196:197]\n" \
    "v_pk_mul_f32 v[28:29], %[inv], v[216:217]\n" \
    "v_pk_mul_f32 v[32:33], %[inv], v[236:237]\n" \
    "v_pk_mul_f32 v[22:23], %[inv], v[178:179]\n" \
    "v_pk_mul_f32 v[26:27], %[inv], v[198:199]\n" \
    "v_pk_mul_f32 v[30:31], %[inv], v[218:219]\n" \
    "v_pk_mul_f32 v[34:35], %[inv], v[238:239]\n" \
    "v_cvt_pk_u8_f32 v36, v20, 0, 0\n" \
    "v_cvt_pk_u8_f32 v37, v24, 0, 0\n" \
    "v_cvt_pk_u8_f32 v38, v28, 0, 0\n" \
    "v_cvt_pk_u8_f32 v39, v32, 0, 0\n" \
    "v_cvt_pk_u8_f32 v36, v21, 1, v36\n" \
    "v_cvt_pk_u8_f32 v37, v25, 1, v37\n" \
    "v_cvt_pk_u8_f32 v38, v29, 1, v38\n" \
    "v_cvt_pk_u8_f32 v39, v33, 1, v39\n" \
    "v_cvt_pk_u8_f32 v36, v22, 2, v36\n" \
    "v_cvt_pk_u8_f32 v37, v26, 2, v37\n" \
    "v_cvt_pk_u8_f32 v38, v30, 2, v38\n" \
    "v_cvt_pk_u8_f32 v39, v34, 2, v39\n" \
    "v_cvt_pk_u8_f32 v36, v23, 3, v36\n" \
    "v_cvt_pk_u8_f32 v37, v27, 3, v37\n" \
    "v_cvt_pk_u8_f32 v38, v31, 3, v38\n" \
    "v_cvt_pk_u8_f32 v39, v35, 3, v39\n" \
    "s_mov_b64 exec, %[odd]\n" \
    "v_swap_b32 v36, v37\n" \
    "v_swap_b32 v38, v39\n" \
    "s_mov_b64 exec, -1\n" \
    "global_store_dwordx4 %[o0], v[36:39], off offset:64 sc1\n" \
    "v_pk_mul_f32 v[0:1], %[inv], v[100:101]\n" \
    "v_pk_mul_f32 v[4:5], %[inv], v[120:121]\n" \
    "v_pk_mul_f32 v[8:9], %[inv], v[140:141]\n" \
    "v_pk_mul_f32 v[12:13], %[inv], v[160:161]\n" \
    "v_pk_mul_f32 v[2:3], %[inv], v[102:103]\n" \
    "v_pk_mul_f32 v[6:7], %[inv], v[122:123]\n" \
    "v_pk_mul_f32 v[10:11], %[inv], v[142:143]\n" \
    "v_pk_mul_f32 v[14:15], %[inv], v[162:163]\n" \
    "v_cvt_pk_u8_f32 v16, v0, 0, 0\n" \
    "v_cvt_pk_u8_f32 v17, v4, 0, 0\n" \
    "v_cvt_pk_u8_f32 v18, v8, 0, 0\n" \
    "v_cvt_pk_u8_f32 v19, v12, 0, 0\n" \
    "v_cvt_pk_u8_f32 v16, v1, 1, v16\n" \
    "v_cvt_pk_u8_f32 v17, v5, 1, v17\n" \
    "v_cvt_pk_u8_f32 v18, v9, 1, v18\n" \
    "v_cvt_pk_u8_f32 v19, v13, 1, v19\n" \
    "v_cvt_pk_u8_f32 v16, v2, 2, v16\n" \
    "v_cvt_pk_u8_f32 v17, v6, 2, v17\n" \
    "v_cvt_pk_u8_f32 v18, v10, 2, v18\n" \
    "v_cvt_pk_u8_f32 v19, v14, 2, v19\n" \
    "v_cvt_pk_u8_f32 v16, v3, 3, v16\n" \
    "v_cvt_pk_u8_f32 v17, v7, 3, v17\n" \
    "v_cvt_pk_u8_f32 v18, v11, 3, v18\n" \
    "v_cvt_pk_u8_f32 v19, v15, 3, v19\n" \
    "s_mov_b64 exec, %[odd]\n" \
    "v_swap_b32 v16, v17\n" \
    "v_swap_b32 v18, v19\n" \
    "s_mov_b64 exec, -1\n" \
    "global_store_dwordx4 %[o1], v[16:19], off offset:0 sc1\n" \
    "v_pk_mul_f32 v[20:21], %[inv], v[180:181]\n" \
    "v_pk_mul_f32 v[24:25], %[inv], v[200:201]\n" \
    "v_pk_mul_f32 v[28:29], %[inv], v[220:221]\n" \
    "v_pk_mul_f32 v[32:33], %[inv], v[240:241]\n" \
    "v_pk_mul_f32 v[22:23], %[inv], v[182:183]\n" \
    "v_pk_mul_f32 v[26:27], %[inv], v[202:203]\n" \
    "v_pk_mul_f32 v[30:31], %[inv], v[222:223]\n" \
    "v_pk_mul_f32 v[34:35], %[inv], v[242:243]\n" \
    "v_cvt_pk_u8_f32 v36, v20, 0, 0\n" \
    "v_cvt_pk_u8_f32 v37, v24, 0, 0\n" \
    "v_cvt_pk_u8_f32 v38, v28, 0, 0\n" \
    "v_cvt_pk_u8_f32 v39, v32, 0, 0\n" \
    "v_cvt_pk_u8_f32 v36, v21, 1, v36\n" \
    "v_cvt_pk_u8_f32 v37, v25, 1, v37\n" \
    "v_cvt_pk_u8_f32 v38, v29, 1, v38\n" \
    "v_cvt_pk_u8_f32 v39, v33, 1, v39\n" \
    "v_cvt_pk_u8_f32 v36, v22, 2, v36\n" \
    "v_cvt_pk_u8_f32 v37, v26, 2, v37\n" \
    "v_cvt_pk_u8_f32 v38, v30, 2, v38\n" \
    "v_cvt_pk_u8_f32 v39, v34, 2, v39\n" \
    "v_cvt_pk_u8_f32 v36, v23, 3, v36\n" \
    "v_cvt_pk_u8_f32 v37, v27, 3, v37\n" \
    "v_cvt_pk_u8_f32 v38, v31, 3, v38\n" \
    "v_cvt_pk_u8_f32 v39, v35, 3, v39\n" \
    "s_mov_b64 exec, %[odd]\n" \
    "v_swap_b32 v36, v37\n" \
    "v_swap_b32 v38, v39\n" \
    "s_mov_b64 exec, -1\n" \
    "global_store_dwordx4 %[o1], v[36:39], off offset:64 sc1\n" \
    "v_pk_mul_f32 v[0:1], %[inv], v[104:105]\n" \
    "v_pk_mul_f32 v[4:5], %[inv], v[124:125]\n" \
    "v_pk_mul_f32 v[8:9], %[inv], v[144:145]\n" \
    "v_pk_mul_f32 v[12:13], %[inv], v[164:165]\n" \
    "v_pk_mul_f32 v[2:3], %[inv], v[106:107]\n" \
    "v_pk_mul_f32 v[6:7], %[inv], v[126:127]\n" \
    "v_pk_mul_f32 v[10:11], %[inv], v[146:147]\n" \
    "v_pk_mul_f32 v[14:15], %[inv], v[166:167]\n" \
    "v_cvt_pk_u8_f32 v16, v0, 0, 0\n" \
    "v_cvt_pk_u8_f32 v17, v4, 0, 0\n" \
    "v_cvt_pk_u8_f32 v18, v8, 0, 0\n" \
    "v_cvt_pk_u8_f32 v19, v12, 0, 0\n" \
    "v_cvt_pk_u8_f32 v16, v1, 1, v16\n" \
    "v_cvt_pk_u8_f32 v17, v5, 1, v17\n" \
    "v_cvt_pk_u8_f32 v18, v9, 1, v18\n" \
    "v_cvt_pk_u8_f32 v19, v13, 1, v19\n" \
    "v_cvt_pk_u8_f32 v16, v2, 2, v16\n" \
    "v_cvt_pk_u8_f32 v17, v6, 2, v17\n" \
    "v_cvt_pk_u8_f32 v18, v10, 2, v18\n" \
    "v_cvt_pk_u8_f32 v19, v14, 2, v19\n" \
    "v_cvt_pk_u8_f32 v16, v3, 3, v16\n" \
    "v_cvt_pk_u8_f32 v17, v7, 3, v17\n" \
    "v_cvt_pk_u8_f32 v18, v11, 3, v18\n" \
    "v_cvt_pk_u8_f32 v19, v15, 3, v19\n" \
    "s_mov_b64 exec, %[odd]\n" \
    "v_swap_b32 v16, v17\n" \
    "v_swap_b32 v18, v19\n" \
    "s_mov_b64 exec, -1\n" \
    "global_store_dwordx4 %[o2], v[16:19], off offset:0 sc1\n" \
    "v_pk_mul_f32 v[20:21], %[inv], v[184:185]\n" \
    "v_pk_mul_f32 v[24:25], %[inv], v[204:205]\n" \
    "v_pk_mul_f32 v[28:29], %[inv], v[224:225]\n" \
    "v_pk_mul_f32 v[32:33], %[inv], v[244:245]\n" \
    "v_pk_mul_f32 v[22:23], %[inv], v[186:187]\n" \
    "v_pk_mul_f32 v[26:27], %[inv], v[206:207]\n" \
    "v_pk_mul_f32 v[30:31], %[inv], v[226:227]\n" \
    "v_pk_mul_f32 v[34:35], %[inv], v[246:247]\n" \
    "v_cvt_pk_u8_f32 v36, v20, 0, 0\n" \
    "v_cvt_pk_u8_f32 v37, v24, 0, 0\n" \
    "v_cvt_pk_u8_f32 v38, v28, 0, 0\n" \
    "v_cvt_pk_u8_f32 v39, v32, 0, 0\n" \
    "v_cvt_pk_u8_f32 v36, v21, 1, v36\n" \
    "v_cvt_pk_u8_f32 v37, v25, 1, v37\n" \
    "v_cvt_pk_u8_f32 v38, v29, 1, v38\n" \
    "v_cvt_pk_u8_f32 v39, v33, 1, v39\n" \
    "v_cvt_pk_u8_f32 v36, v22, 2, v36\n" \
    "v_cvt_pk_u8_f32 v37, v26, 2, v37\n" \
    "v_cvt_pk_u8_f32 v38, v30, 2, v38\n" \
    "v_cvt_pk_u8_f32 v39, v34, 2, v39\n" \
    "v_cvt_pk_u8_f32 v36, v23, 3, v36\n" \
    "v_cvt_pk_u8_f32 v37, v27, 3, v37\n" \
    "v_cvt_pk_u8_f32 v38, v31, 3, v38\n" \
    "v_cvt_pk_u8_f32 v39, v35, 3, v39\n" \
    "s_mov_b64 exec, %[odd]\n" \
    "v_swap_b32 v36, v37\n" \
    "v_swap_b32 v38, v39\n" \
    "s_mov_b64 exec, -1\n" \
    "global_store_dwordx4 %[o2], v[36:39], off offset:64 sc1\n" \
    "v_pk_mul_f32 v[0:1], %[inv], v[108:109]\n" \
    "v_pk_mul_f32 v[4:5], %[inv], v[128:129]\n" \
    "v_pk_mul_f32 v[8:9], %[inv], v[148:149]\n" \
    "v_pk_mul_f32 v[12:13], %[inv], v[168:169]\n" \
    "v_pk_mul_f32 v[2:3], %[inv], v[110:111]\n" \
    "v_pk_mul_f32 v[6:7], %[inv], v[130:131]\n" \
    "v_pk_mul_f32 v[10:11], %[inv], v[150:151]\n" \
    "v_pk_mul_f32 v[14:15], %[inv], v[170:171]\n" \
    "v_cvt_pk_u8_f32 v16, v0, 0, 0\n" \
    "v_cvt_pk_u8_f32 v17, v4, 0, 0\n" \
    "v_cvt_pk_u8_f32 v18, v8, 0, 0\n" \
    "v_cvt_pk_u8_f32 v19, v12, 0, 0\n" \
    "v_cvt_pk_u8_f32 v16, v1, 1, v16\n" \
    "v_cvt_pk_u8_f32 v17, v5, 1, v17\n" \
    "v_cvt_pk_u8_f32 v18, v9, 1, v18\n" \
    "v_cvt_pk_u8_f32 v19, v13, 1, v19\n" \
    "v_cvt_pk_u8_f32 v16, v2, 2, v16\n" \
    "v_cvt_pk_u8_f32 v17, v6, 2, v17\n" \
    "v_cvt_pk_u8_f32 v18, v10, 2, v18\n" \
    "v_cvt_pk_u8_f32 v19, v14, 2, v19\n" \
    "v_cvt_pk_u8_f32 v16, v3, 3, v16\n" \
    "v_cvt_pk_u8_f32 v17, v7, 3, v17\n" \
    "v_cvt_pk_u8_f32 v18, v11, 3, v18\n" \
    "v_cvt_pk_u8_f32 v19, v15, 3, v19\n" \
    "s_mov_b64 exec, %[odd]\n" \
    "v_swap_b32 v16, v17\n" \
    "v_swap_b32 v18, v19\n" \
    "s_mov_b64 exec, -1\n" \
    "global_store_dwordx4 %[o3], v[16:19], off offset:0 sc1\n" \
    "v_pk_mul_f32 v[20:21], %[inv], v[188:189]\n" \
    "v_pk_mul_f32 v[24:25], %[inv], v[208:209]\n" \
    "v_pk_mul_f32 v[28:29], %[inv], v[228:229]\n" \
    "v_pk_mul_f32 v[32:33], %[inv], v[248:249]\n" \
    "v_pk_mul_f32 v[22:23], %[inv], v[190:191]\n" \
    "v_pk_mul_f32 v[26:27], %[inv], v[210:211]\n" \
    "v_pk_mul_f32 v[30:31], %[inv], v[230:231]\n" \
    "v_pk_mul_f32 v[34:35], %[inv], v[250:251]\n" \
    "v_cvt_pk_u8_f32 v36, v20, 0, 0\n" \
    "v_cvt_pk_u8_f32 v37, v24, 0, 0\n" \
    "v_cvt_pk_u8_f32 v38, v28, 0, 0\n" \
    "v_cvt_pk_u8_f32 v39, v32, 0, 0\n" \
    "v_cvt_pk_u8_f32 v36, v21, 1, v36\n" \
    "v_cvt_pk_u8_f32 v37, v25, 1, v37\n" \
    "v_cvt_pk_u8_f32 v38, v29, 1, v38\n" \
    "v_cvt_pk_u8_f32 v39, v33, 1, v39\n" \
    "v_cvt_pk_u8_f32 v36, v22, 2, v36\n" \
    "v_cvt_pk_u8_f32 v37, v26, 2, v37\n" \
    "v_cvt_pk_u8_f32 v38, v30, 2, v38\n" \
    "v_cvt_pk_u8_f32 v39, v34, 2, v39\n" \
    "v_cvt_pk_u8_f32 v36, v23, 3, v36\n" \
    "v_cvt_pk_u8_f32 v37, v27, 3, v37\n" \
    "v_cvt_pk_u8_f32 v38, v31, 3, v38\n" \
    "v_cvt_pk_u8_f32 v39, v35, 3, v39\n" \
    "s_mov_b64 exec, %[odd]\n" \
    "v_swap_b32 v36, v37\n" \
    "v_swap_b32 v38, v39\n" \
    "s_mov_b64 exec, -1\n" \
    "global_store_dwordx4 %[o3], v[36:39], off offset:64 sc1\n" \
    "s_cmp_lt_u32 %[nm], 5\n" \
    "s_cbranch_scc1 Lq%=\n" \
    "v_pk_mul_f32 v[0:1], %[inv], v[112:113]\n" \
    "v_pk_mul_f32 v[4:5], %[inv], v[132:133]\n" \
    "v_pk_mul_f32 v[8:9], %[inv], v[152:153]\n" \
    "v_pk_mul_f32 v[12:13], %[inv], v[172:173]\n" \
    "v_pk_mul_f32 v[2:3], %[inv], v[114:115]\n" \
    "v_pk_mul_f32 v[6:7], %[inv], v[134:135]\n" \
    "v_pk_mul_f32 v[10:11], %[inv], v[154:155]\n" \
    "v_pk_mul_f32 v[14:15], %[inv], v[174:175]\n" \
    "v_cvt_pk_u8_f32 v16, v0, 0, 0\n" \
    "v_cvt_pk_u8_f32 v17, v4, 0, 0\n" \
    "v_cvt_pk_u8_f32 v18, v8, 0, 0\n" \
    "v_cvt_pk_u8_f32 v19, v12, 0, 0\n" \
    "v_cvt_pk_u8_f32 v16, v1, 1, v16\n" \
    "v_cvt_pk_u8_f32 v17, v5, 1, v17\n" \
    "v_cvt_pk_u8_f32 v18, v9, 1, v18\n" \
    "v_cvt_pk_u8_f32 v19, v13, 1, v19\n" \
    "v_cvt_pk_u8_f32 v16, v2, 2, v16\n" \
    "v_cvt_pk_u8_f32 v17, v6, 2, v17\n" \
    "v_cvt_pk_u8_f32 v18, v10, 2, v18\n" \
    "v_cvt_pk_u8_f32 v19, v14, 2, v19\n" \
    "v_cvt_pk_u8_f32 v16, v3, 3, v16\n" \
    "v_cvt_pk_u8_f32 v17, v7, 3, v17\n" \
    "v_cvt_pk_u8_f32 v18, v11, 3, v18\n" \
    "v_cvt_pk_u8_f32 v19, v15, 3, v19\n" \
    "s_mov_b64 exec, %[odd]\n" \
    "v_swap_b32 v16, v17\n" \
    "v_swap_b32 v18, v19\n" \
    "s_mov_b64 exec, -1\n" \
    "global_store_dwordx4 %[o4], v[16:19], off offset:0 sc1\n" \
    "v_pk_mul_f32 v[20:21], %[inv], v[192:193]\n" \
    "v_pk_mul_f32 v[24:25], %[inv], v[212:213]\n" \
    "v_pk_mul_f32 v[28:29], %[inv], v[232:233]\n" \
    "v_pk_mul_f32 v[32:33], %[inv], v[252:253]\n" \
    "v_pk_mul_f32 v[22:23], %[inv], v[194:195]\n" \
    "v_pk_mul_f32 v[26:27], %[inv], v[214:215]\n" \
    "v_pk_mul_f32 v[30:31], %[inv], v[234:235]\n" \
    "v_pk_mul_f32 v[34:35], %[inv], v[254:255]\n" \
    "v_cvt_pk_u8_f32 v36, v20, 0, 0\n" \
    "v_cvt_pk_u8_f32 v37, v24, 0, 0\n" \
    "v_cvt_pk_u8_f32 v38, v28, 0, 0\n" \
    "v_cvt_pk_u8_f32 v39, v32, 0, 0\n" \
    "v_cvt_pk_u8_f32 v36, v21, 1, v36\n" \
    "v_cvt_pk_u8_f32 v37, v25, 1, v37\n" \
    "v_cvt_pk_u8_f32 v38, v29, 1, v38\n" \
    "v_cvt_pk_u8_f32 v39, v33, 1, v39\n" \
    "v_cvt_pk_u8_f32 v36, v22, 2, v36\n" \
    "v_cvt_pk_u8_f32 v37, v26, 2, v37\n" \
    "v_cvt_pk_u8_f32 v38, v30, 2, v38\n" \
    "v_cvt_pk_u8_f32 v39, v34, 2, v39\n" \
    "v_cvt_pk_u8_f32 v36, v23, 3, v36\n" \
    "v_cvt_pk_u8_f32 v37, v27, 3, v37\n" \
    "v_cvt_pk_u8_f32 v38, v31, 3, v38\n" \
    "v_cvt_pk_u8_f32 v39, v35, 3, v39\n" \
    "s_mov_b64 exec, %[odd]\n" \
    "v_swap_b32 v36, v37\n" \
    "v_swap_b32 v38, v39\n" \
    "s_mov_b64 exec, -1\n" \
    "global_store_dwordx4 %[o4], v[36:39], off offset:64 sc1\n" \
    "Lq%=:\n" \
    "s_nop 1\n" \
    ""
#define QUANT_CLOBBERS "v0", "v1", "v2", "v3", "v4", "v5", "v6", "v7", "v8", "v9", "v10", "v11", "v12", "v13", "v14", "v15", "v16", "v17", "v18", "v19", "v20", "v21", "v22", "v23", "v24", "v25", "v26", "v27", "v28", "v29", "v30", "v31", "v32", "v33", "v34", "v35", "v36", "v37", "v38", "v39"

__global__ void __launch_bounds__(512, 2) ag_gemm_kernel(Params P) {
    __shared__ __attribute__((aligned(1024))) char smem[LDS_BYTES];
    char* const ws = P.rot[0];
    asm volatile("" :: "s"(P.x), "s"(P.w), "s"(P.out), "s"(P.rank), "s"(ws));
    const int b = blockIdx.x;
    const int wave = rfl(threadIdx.x >> 6);
    u32 lane = threadIdx.x & 63;
    const int xc = b & 7, idx = b >> 3;
    const bool gemm_wg = idx < NGEMM_PER;
    const int cls = gemm_wg ? 0 : xc;

    u32x4 d[4];
    const u32 pu = (wave - 2) * NCOMM_PER + (idx - NGEMM_PER);
    const bool pusher = !gemm_wg && wave >= 2 && pu < 32 && RAWKB > 0;
    const bool two = b >= PF_END;
    const bool packer = gemm_wg && (wave < 4 || two);
    const u32 quad = wave < 4 ? b : b + (NQUAD - 8 * NGEMM_PER), w4 = wave & 3;
    const u32 eg = pusher ? 0 : G0 + (quad >> 3), emb = pusher ? pu : (quad & 7) * 4 + w4;
    if (pusher || packer) {
        const char* src = P.x + (size_t)(emb * 16 + (lane & 15)) * (GK * 2) + eg * 256 + (lane >> 4) * 16;
#pragma unroll
        for (int i = 0; i < 4; ++i) d[i] = *(const GAS u32x4*)(src + i * 64);
    }
    char* const sws = P.rot[cls];
    char* lws = P.rot[lane & 7];
    const u32 callv = ld_agent(ws + OFF_CNT + b * 128);
    u32 sink0 = 0;
    if (gemm_wg && wave == 5 && b >= EARLY_WG && b < PF_END) {
        u64 pc; u32 len;
        asm volatile("s_getpc_b64 %0\nag_pc:\n\ts_mov_b32 %1, ag_e0-ag_pc\n\ts_max_i32 %1, %1, ag_e1-ag_pc\n\ts_max_i32 %1, %1, ag_e2-ag_pc\n\ts_max_i32 %1, %1, 0"
                     : "=&s"(pc), "=&s"(len) :: "scc");
#pragma unroll
        for (int i = 0; i < 5; ++i) asm volatile("global_load_dword %0, %1, off" : "+v"(sink0) : "v"((const char*)pc + min(i * 8192 + lane * 128, len)) : "memory");
    }
    lds_st((u32)(size_t)(LAS char*)smem + threadIdx.x * 4, 0);
    if (!gemm_wg && wave >= 4) lds_st(L_LUT + (threadIdx.x - 256) * 4, lut_entry(threadIdx.x - 256));
    if (gemm_wg && wave == 7 && b >= EARLY_WG && b < PF_END) {
        u32 wsink = 0;
        const u32 pw = b - (EARLY_WG);
        constexpr u32 NPW = PF_END - (EARLY_WG);
        const char* wp = P.w + lane * 128;
        u32 i = WFROM * 16 + pw;
        __builtin_amdgcn_s_sleep(PF_DELAY);
        for (u32 k = 0; k < PF_PRE && i < WBURST * 16; ++k, i += NPW)
            asm volatile("global_load_dword %0, %1, off" : "+v"(wsink) : "v"(wp + (size_t)i * 8192) : "memory");
        asm volatile("s_waitcnt lgkmcnt(0)\n\ts_barrier" ::: "memory");
        for (; i < WBURST * 16; i += NPW)
            asm volatile("global_load_dword %0, %1, off" : "+v"(wsink) : "v"(wp + (size_t)i * 8192) : "memory");
        for (i = 5 + pw; i < 128; i += NPW)
            asm volatile("global_load_dword %0, %1, off" : "+v"(wsink) : "v"(ws + OFF_XFLAG + (i * 64 + lane) * 128) : "memory");
        asm volatile("s_waitcnt vmcnt(0)" : "+v"(wsink) :: "memory");
        asm volatile("s_endpgm");
    }
    constexpr u32 L_XCH = L_RING + (RING - 4) * SLOT;
    u32 xw = L_XCH + (wave < 4 ? 16384 : 0) + w4 * 1024 + lane * 16, xr = L_XCH + (wave < 4 ? 16384 : 0) + w4 * 4096 + lane * 16;
    u32 scr = (wave < 4 ? L_ESCR : L_RING) + w4 * ESCR;
    char* xg = sws + OFF_XG + (size_t)(eg * 4) * XPITCH + (size_t)(P.rank * 32 + emb) * 1024 + lane * 16;
    char* xg1 = xg + XPITCH; char* xg2 = xg + 2 * XPITCH; char* xg3 = xg + 3 * XPITCH;
    char* pk16 = ws + OFF_PK + (size_t)((eg * 4 + w4) * 8 + (quad & 7)) * PK_STRIDE + lane * 16;
    char* pk8 = pk16 - lane * 8;
    u32 fo = pusher ? (u32)OFF_RAWF + (P.rank * 32 + pu) * 128
                    : lane < 8 ? (u32)OFF_RFLAG + (P.rank * 256 + quad) * 128 : (u32)OFF_XFLAG + ((eg * 4 + (lane >> 3) - 1) * 64 + P.rank * 8 + (quad & 7)) * 128;
    asm volatile("" : "+v"(xw), "+v"(xr), "+s"(scr), "+v"(xg), "+v"(xg1), "+v"(xg2), "+v"(xg3), "+v"(pk16), "+v"(pk8), "+v"(fo));
    __builtin_amdgcn_s_waitcnt(0x0070);
    asm volatile("s_waitcnt vmcnt(0)" : "+v"(sink0) :: "memory");
    u32 B1 = 0;
    if (packer) {
#pragma unroll
        for (int i = 0; i < 4; ++i) *(LAS u32x4*)(xw + i * 4096) = d[i];
        B1 = ref_exponent(d[0]);
    }
    if (pusher) {
        gst4_sys(xg, d[0]);
        if (RAWKB > 1) gst4_sys(xg1, d[1]);
        if (RAWKB > 2) gst4_sys(xg2, d[2]);
        if (RAWKB > 3) gst4_sys(xg3, d[3]);
    }
    asm volatile("s_waitcnt lgkmcnt(0)\n\ts_barrier" : "+v"(B1) :: "memory");
    const u32 call = rfl(callv) + 1;
    if (packer) {
        u32x4 e[4];
#pragma unroll
        for (int i = 0; i < 4; ++i) e[i] = *(LAS u32x4*)(xr + i * 1024);
        gst4_sys(xg, d[0]); gst4_sys(xg1, d[1]); gst4_sys(xg2, d[2]); gst4_sys(xg3, d[3]);
        encode_block(e, B1, lane, scr, pk16, pk8);
        char* fp = lws + fo;
        u32 old;
        asm volatile("s_waitcnt vmcnt(0)\n\ts_mov_b64 exec, 1\n\tds_add_rtn_u32 %0, %2, %3\n\ts_mov_b64 exec, -1\n\ts_waitcnt lgkmcnt(0)"
                     : "=&v"(old), "+v"(fp) : "v"(wave < 4 ? (u32)L_ECNT : (u32)L_ECNT2), "v"(1u) : "memory");
        if (rfl(old) == 3) {
            if ((0x1010101feull >> lane) & 1) gst1_sys(fp, call);
        }
    }
    if (threadIdx.x == 256) st_agent(ws + OFF_CNT + b * 128, call);
    if (pusher) {
        waitvm<0>();
        if (lane == 0) gst1_sys(sws + fo, call);
    }
    if (!gemm_wg) {
        if (cls != 0) {
            const u32 src = (u32)(P.rank + cls) & 7u;
            const u32 ci = idx - NGEMM_PER;
            const u32 nblk = (1024 - NRAW - ci + NCOMM_PER - 1) / NCOMM_PER;
            static_assert(CRING % 4 == 0, "");
            if (wave == 0) {
                __builtin_amdgcn_s_setprio(3);
                const char* pk = sws + OFF_PK + (size_t)(NRAW + ci) * PK_STRIDE + lane * 16;
                u32 rdy = 0, slot = 0, round = 0;
                for (u32 k = 0; k < (1u << 16); ++k) {
                    bool ok = true;
                    if (lane < 4) { const char* fp = ws + OFF_RFLAG + (src * 256 + quad_of(NRAW + ci + lane * NCOMM_PER)) * 128; ok = (int)(ld_sys(fp) - call) >= 0; }
                    if (__all(ok)) { rdy = 4; break; }
                }
                const u64 tf = rt();
                for (u32 n = 0; n + 4 <= nblk; n += 4) {
                    if (n + 4 > rdy || (round && !__all(lds_ld(L_FREE + (slot + (lane & 3)) * 4) >= round))) {
                        for (u32 k = 0;; ++k) {
                            rdy = rfl(lds_ld(L_RDY));
                            if (n + 4 <= rdy && __all(lds_ld(L_FREE + (slot + (lane & 3)) * 4) >= round)) break;
                            if (k == 0) { waitvm<0>(); lds_st(L_LAND, n); }
                            if (rfl(lds_ld(L_ABORT)) || k >= SPIN_CAP) { st_agent(ws + OFF_TMO, call); lds_st(L_ABORT, 1); return; }
                            __builtin_amdgcn_s_sleep(1);
                        }
                    }
                    u32 keep; int lv = (int)n - (INFL - 1);
#define FETCH1(PK) "s_waitcnt vmcnt(%c[w])\n\t" \
                   "global_load_lds_dwordx4 " PK ", off sc0 sc1\n\t" \
                   "global_load_lds_dwordx4 " PK ", off offset:1024 sc0 sc1\n\t" \
                   "s_mov_b32 exec_hi, " TAILMASK "\n\t" \
                   "global_load_lds_dwordx4 " PK ", off offset:2048 sc0 sc1\n\t" \
                   "s_mov_b64 exec, 1\n\t" \
                   "ds_max_i32 %[la], %[lv]\n\t" \
                   "s_mov_b64 exec, -1\n\t" \
                   "s_add_u32 m0, m0, 3072\n\t" \
                   "v_add_u32 %[lv], 1, %[lv]\n\t"
                    if ((u32)(rt() - tf) * 10 > n * PACE + PACE0) {
                        --lv;
                    asm volatile("s_mov_b32 %[k], m0\n\ts_mov_b32 m0, %[l]\n\t"
                                 FETCH1("%[p0]") FETCH1("%[p1]") FETCH1("%[p2]") FETCH1("%[p3]")
                                 "s_mov_b32 m0, %[k]"
                                 : [k] "=&s"(keep), [lv] "+v"(lv)
                                 : [p0] "v"(pk), [p1] "v"(pk + NCOMM_PER * PK_STRIDE), [p2] "v"(pk + 2 * NCOMM_PER * PK_STRIDE), [p3] "v"(pk + 3 * NCOMM_PER * PK_STRIDE),
                                   [l] "s"(rfl(L_RING + slot * 3072)), [w] "n"(3 * INFL), [la] "v"((u32)L_LAND) : "scc", "memory");
                    } else
                    asm volatile("s_mov_b32 %[k], m0\n\ts_mov_b32 m0, %[l]\n\t"
                                 FETCH1("%[p0]") FETCH1("%[p1]") FETCH1("%[p2]") FETCH1("%[p3]")
                                 "s_mov_b32 m0, %[k]"
                                 : [k] "=&s"(keep), [lv] "+v"(lv)
                                 : [p0] "v"(pk), [p1] "v"(pk + NCOMM_PER * PK_STRIDE), [p2] "v"(pk + 2 * NCOMM_PER * PK_STRIDE), [p3] "v"(pk + 3 * NCOMM_PER * PK_STRIDE),
                                   [l] "s"(rfl(L_RING + slot * 3072)), [w] "n"(3 * (INFL - 1)), [la] "v"((u32)L_LAND) : "scc", "memory");
                    pk += (size_t)4 * NCOMM_PER * PK_STRIDE;
                    slot += 4;
                    if (slot == CRING) { slot = 0; ++round; }
                }
                for (u32 n = nblk & ~3u; n < nblk; ++n) {
                    for (u32 k = 0; rfl(lds_ld(L_RDY)) <= n || rfl(lds_ld(L_FREE + slot * 4)) < round; ++k) {
                        if (k == 0) { waitvm<0>(); lds_st(L_LAND, n); }
                        if (rfl(lds_ld(L_ABORT)) || k >= SPIN_CAP) { st_agent(ws + OFF_TMO, call); lds_st(L_ABORT, 1); return; }
                    }
                    u32 keep; int lv = (int)n - (INFL - 1);
                    asm volatile("s_mov_b32 %[k], m0\n\ts_mov_b32 m0, %[l]\n\t" FETCH1("%[p0]") "s_mov_b32 m0, %[k]"
                                 : [k] "=&s"(keep), [lv] "+v"(lv)
                                 : [p0] "v"(pk), [l] "s"(rfl(L_RING + slot * 3072)), [w] "n"(3 * (INFL - 1)), [la] "v"((u32)L_LAND) : "scc", "memory");
                    pk += (size_t)NCOMM_PER * PK_STRIDE; ++slot;
                }
#pragma unroll
                for (int k = INFL; k >= 0; --k) {
                    asm volatile("s_waitcnt vmcnt(%c0)" :: "n"(3 * k) : "memory");
                    if (lane == 0) asm volatile("ds_max_i32 %0, %1" :: "v"((u32)L_LAND), "v"((int)nblk - k) : "memory");
                }
                return;
            }
            if (wave == 1) {
                const u64 t0 = rt();
                u32 rq = 0;
                for (u32 it = 0; rq < NQUAD; ++it) {
                    const u32 width = rq < 16 ? 16u : 64u;
                    bool ok = true;
                    if (lane < width) { const char* fp = ws + OFF_RFLAG + (src * 256 + min(rq + lane, (u32)NQUAD - 1)) * 128; ok = (int)(ld_sys(fp) - call) >= 0; }
                    const u64 mk = __ballot(ok);
                    const u32 adv = (mk == ~0ull ? width : (u32)__builtin_ctzll(~mk)) & ~7u;
                    if (adv) { rq += adv; lds_st(L_RDY, (rq * 4 + 32 * G0 - NRAW - ci + NCOMM_PER - 1) / NCOMM_PER); continue; }
                    if ((it & 15) == 15 && (rfl(ld_agent(ws + OFF_TMO)) == call || rt() - t0 > DEADLINE)) {
                        st_agent(ws + OFF_TMO, call); lds_st(L_ABORT, 1); return;
                    }
                    __builtin_amdgcn_s_sleep(1);
                }
            }
            const u32 strm = L_STRM + wave * STRM;
            char* pflag = nullptr;
            const u32 a16 = lane * 16, a8 = lane * 8, ac = (lane >> 1) * 12 + (lane & 1) * 4, ad = (lane >> 1) * 12 + (lane & 1) * 2, ap = lane * 4;
            u32 curB = 0, T1 = 0, T2l = 0, T2h = 0;
            for (;;) {
                u32 n, land;
                asm volatile("s_mov_b64 exec, 1\n\tds_add_rtn_u32 %0, %2, %3 offset:8\n\tds_read_b32 %1, %2\n\ts_mov_b64 exec, -1\n\ts_waitcnt lgkmcnt(0)"
                             : "=&v"(n), "=&v"(land) : "v"((u32)L_LAND), "v"(1u) : "memory");
                n = rfl(n);
                const u32 i = NRAW + ci + n * NCOMM_PER, rb = src * 8 + (i & 7);
                const u32 xo = (u32)OFF_XG + (i >> 3) * XPITCH + rb * 4096;
                const u32 slot = n % CRING, stage = L_RING + slot * 3072;
                if (rfl(land) <= n) {
                    if (pflag) { publish(pflag, call, lane); pflag = nullptr;
                    }
                    if (n >= nblk) break;
                    bool ok = false;
#pragma unroll 1
                    for (u32 k = 0; k < SPIN_CAP; ++k) {
                        u32x2 al;
                        asm volatile("ds_read2_b32 %0, %1 offset1:4\n\ts_waitcnt lgkmcnt(0)" : "=&v"(al) : "v"((u32)L_ABORT) : "memory");
                        static_assert(L_LAND == L_ABORT + 16, "");
                        if (rfl(al[1]) > n) { ok = true; break; }
                        if (rfl(al[0])) break;
                        __builtin_amdgcn_s_sleep(1);
                    }
                    if (!ok) { st_agent(ws + OFF_TMO, call); lds_st(L_ABORT, 1); return; }
                }
                Packed k;
                read_packed(k, a16, a8, ac, ad, ap, stage);
                const u32 hdr = rfl(k.hp);
                if (__builtin_expect((hdr & 0xc00000ffu) != curB, 0)) {
                    curB = hdr & 255u;
                    const u32 B4 = curB * 0x01010101u;
                    T1 = rot8x4(B4 - 0x03020100u); T2l = rot8x4(B4 - 0x0a090800u); T2h = rot8x4(B4 - 0x07060504u);
                    if (hdr & 0x40000000u) {
                        if (lane < 16) *(LAS u32x4*)(stage + 2816 + a16) = gld4_sys_wait(sws + OFF_PK + (size_t)i * PK_STRIDE + 2816 + a16);
                        read_packed(k, a16, a8, ac, ad, ap + 256, stage);
                    }
                }
                if (lane == 0) lds_st(L_FREE + slot * 4, n / CRING + 1);
                if ((int)hdr >= 0) {
                    decode_block(k, ~(0xffffffffu >> ((hdr >> 8) & 31u)), T1, T2l, T2h, lane, strm, ws + xo + a16, pflag, call);
                } else {
                    if (pflag) { publish(pflag, call, lane); }
                    u32x4 r0, r1, r2, r3;
                    asm volatile("global_load_dwordx4 %0, %4, off sc0 sc1\n\tglobal_load_dwordx4 %1, %4, off offset:1024 sc0 sc1\n\t"
                                 "global_load_dwordx4 %2, %4, off offset:2048 sc0 sc1\n\tglobal_load_dwordx4 %3, %4, off offset:3072 sc0 sc1\n\ts_waitcnt vmcnt(0)\n\t"
                                 "global_store_dwordx4 %5, %0, off sc1\n\tglobal_store_dwordx4 %5, %1, off offset:1024 sc1\n\t"
                                 "global_store_dwordx4 %5, %2, off offset:2048 sc1\n\tglobal_store_dwordx4 %5, %3, off offset:3072 sc1\n\ts_nop 1"
                                 : "=&v"(r0), "=&v"(r1), "=&v"(r2), "=&v"(r3) : "v"(sws + xo + a16), "v"(ws + xo + a16) : "memory");
                }
                pflag = ws + OFF_XFLAG + ((i >> 3) * 64 + rb) * 128;
            }
            asm volatile("ag_e2:");
        } else if (idx != NGEMM_PER) {
            asm volatile("s_waitcnt vmcnt(0)\n\ts_barrier" ::: "memory");
            u32 sink = 0;
            const char* wp = P.w + (size_t)WBURST * 32 * (NL * 2) + lane * 128;
            for (u32 i = (idx - NGEMM_PER - 1) * 8 + wave; i < (128 - WBURST) * 32 * (NL * 2) / 8192; i += 8 * (NCOMM_PER - 1))
                asm volatile("global_load_dword %0, %1, off\n\ts_waitcnt vmcnt(2)" : "+v"(sink) : "v"(wp + (size_t)i * 8192) : "memory");
            asm volatile("s_waitcnt vmcnt(0)" : "+v"(sink) :: "memory");
        }
        if (cls == 0 && idx == NGEMM_PER && wave == 0) {
            u32 l7 = lane & 7; asm volatile("" : "+v"(l7));
            char* aws = P.rot[l7];
            asm volatile("" : "+v"(aws));
            const u64 t0 = rt();
            const char* gp = ws + OFF_AMAXL + lane * 128;
            char* const dst = aws + OFF_AMAXG + P.rank * 128;
            u64 miss[4] = {~0ull, ~0ull, ~0ull, 0xffffull};
            u32 m = 0, last = 0;
            bool go = false;
            for (u32 it = 0;; ++it) {
                u64 v[4] = {0, 0, 0, 0};
                asm volatile("s_mov_b64 exec, %5\n\tglobal_load_dwordx2 %0, %4, off sc0 sc1\n\t"
                             "s_mov_b64 exec, %6\n\tglobal_load_dwordx2 %1, %9, off sc0 sc1\n\t"
                             "s_mov_b64 exec, %7\n\tglobal_load_dwordx2 %2, %10, off sc0 sc1\n\t"
                             "s_mov_b64 exec, %8\n\tglobal_load_dwordx2 %3, %11, off sc0 sc1\n\t"
                             "s_mov_b64 exec, -1\n\ts_waitcnt vmcnt(0)"
                             : "+v"(v[0]), "+v"(v[1]), "+v"(v[2]), "+v"(v[3])
                             : "v"(gp), "s"(miss[0]), "s"(miss[1]), "s"(miss[2]), "s"(miss[3]), "v"(gp + 8192), "v"(gp + 16384), "v"(gp + 24576) : "memory");
                u64 any = 0;
#pragma unroll
                for (int i = 0; i < 4; ++i) {
                    const bool in = (u32)(v[i] >> 32) == call;
                    m = max(m, in ? (u32)v[i] : 0u);
                    const u64 mk = __ballot(in);
                    any |= mk; miss[i] &= ~mk;
                }
                const u32 left = __builtin_popcountll(miss[0]) + __builtin_popcountll(miss[1]) + __builtin_popcountll(miss[2]) + __builtin_popcountll(miss[3]);
                if (any) {
                    if (__any(m > last)) last = wave_maxu(m);
                    if (!left) {
                        const u64 g = ((u64)call << 32) | last;
                        if (lane < NRANK) {
                            st_sys64(dst, g);
                        }
                        break;
                    }
                    if (!go && left <= 8 * NGEMM_PER - SPEC_AT) { go = true; if (lane == 0) st_sys64(ws + OFF_AMAXG + 9 * 128, (u64)call << 32); }
                }
                if ((it & 63) == 63) {
                    if (rfl(ld_agent(ws + OFF_TMO)) == call) break;
                    if (rt() - t0 > DEADLINE) { st_agent(ws + OFF_TMO, call); break; }
                }
                if (left > 8 * NGEMM_PER - 40) __builtin_amdgcn_s_sleep(32);
            }
        }
        return;
    }

    const int lin = (xc >> 1) * NGEMM_PER + idx;
    const int tm = lin >> 3, tn = (xc & 1) * 8 + (lin & 7);
    if (wave < 4) {
        const int nm = tm < 12 ? 5 : 4;
        const int mb0 = tm * 20 + wave * nm;
        f32x32 A0, A1, A2, A3, A4;
        u32 sbad, sacc, mx;
        {
            const u32 g = lane >> 4, q = (lane >> 2) & 3, p = lane & 3;
            u32 a0 = L_RING + g * 1024 + q * 64 + ((p ^ (g & 1)) * 16) + ((p & 1) * 8);
            u32x4 xrs;
            { const u64 xb = (u64)(ws + OFF_XG); xrs[0] = (u32)xb; xrs[1] = (u32)(xb >> 32); xrs[2] = (u32)XG_BYTES; xrs[3] = 0x00020000u; }
            const u32 xoff = (u32)mb0 * 1024 + lane * 16;
            const u32 xoff4 = xoff + (nm == 5 ? 4096u : 0x40000000u);
            u32 sx, sn, sring, stmp, scnt, sxr;
            asm volatile(KLOOP_ASM
                         : "={v[96:127]}"(A0), "={v[128:159]}"(A1), "={v[160:191]}"(A2), "={v[192:223]}"(A3), "={v[224:255]}"(A4), "+{v84}"(a0), "=&{v92}"(mx),
                           [sbad] "=&s"(sbad), [sacc] "=&s"(sacc), [sx] "=&s"(sx), [sn] "=&s"(sn), [sring] "=&s"(sring),
                           [stmp] "=&s"(stmp), [scnt] "=&s"(scnt), [sxr] "=&s"(sxr)
                         : [xoff] "{v86}"(xoff), [xrs] "s"(xrs), [xst] "s"(XPITCH), [dstep] "s"(3 * SLOT), [dwrap] "s"((3 - RING) * SLOT), [xoff4] "{v90}"(xoff4), [actl] "{v87}"((u32)(L_PAIR + wave * 8)),
                           [acn] "{v91}"((u32)(L_CONS + wave * 256 + lane * 4))
                         : KLOOP_CLOBBERS, "scc", "memory");
        }
        lane = __builtin_amdgcn_mbcnt_hi(~0u, __builtin_amdgcn_mbcnt_lo(~0u, 0u));
        if (sbad) { st_agent(ws + OFF_TMO, call); lds_st(L_ABORT, 1); }
        {
            mx = max(mx, (u32)__builtin_amdgcn_update_dpp(0, (int)mx, 0x111, 0xf, 0xf, false));
            mx = max(mx, (u32)__builtin_amdgcn_update_dpp(0, (int)mx, 0x112, 0xf, 0xf, false));
            mx = max(mx, (u32)__builtin_amdgcn_update_dpp(0, (int)mx, 0x114, 0xf, 0xf, false));
            mx = max(mx, (u32)__builtin_amdgcn_update_dpp(0, (int)mx, 0x118, 0xf, 0xf, false));
            if ((lane & 15) == 15) {
                u32 old; u32x2 mab;
                asm volatile("ds_max_u32 %2, %3 offset:4\n\tds_add_rtn_u32 %0, %2, %4 offset:8\n\tds_read_b64 %1, %2\n\ts_waitcnt lgkmcnt(0)"
                             : "=&v"(old), "=&v"(mab) : "v"((u32)L_ABORT), "v"(mx), "v"(1u) : "memory");
                if (old == 15 && mab[0] == 0) st_sys64(ws + OFF_AMAXL + b * 128, ((u64)call << 32) | mab[1]);
            }
        }
        char* orow = P.out + (size_t)(mb0 * 16 + (lane & 15)) * NL + tn * 128 + (lane >> 4) * 16;
        asm volatile("" : "+v"(orow));
        u32 seen = 0, done_inv = 0xffffffffu;
#pragma unroll 1
        while (seen < 2) {
            u32x2 res;
            u32 st = 3;
#pragma unroll 1
            for (u32 k = 0; k < (1u << 25); ++k) {
                asm volatile("ds_read_b64 %0, %1\n\ts_waitcnt lgkmcnt(0)" : "=&v"(res) : "v"((u32)L_RES) : "memory");
                if (rfl(res[0]) > seen || (seen && rfl(res[1]) != done_inv)) { st = rfl(res[0]); break; }
                if (!seen) __builtin_amdgcn_s_sleep(1);
            }
            seen = st;
            if (seen != 3 && rfl(res[1]) == done_inv) continue;
            if (done_inv != 0xffffffffu) waitvm<0>();
            done_inv = rfl(res[1]);
            if (seen == 3) {
                for (int i = 0; i < 2 * nm; ++i) gst4_agent(orow + (size_t)(i >> 1) * 16 * NL + (i & 1) * 64, u32x4{0x80808080u, 0x80808080u, 0x80808080u, 0x80808080u});
            } else {
                asm volatile(QUANT_ASM
                             :: "{v[96:127]}"(A0), "{v[128:159]}"(A1), "{v[160:191]}"(A2), "{v[192:223]}"(A3), "{v[224:255]}"(A4),
                                [inv] "s"(((u64)done_inv << 32) | done_inv), [odd] "s"(0xffff0000ffff0000ull), [nm] "s"(nm),
                                [o0] "v"(orow), [o1] "v"(orow + 16 * NL), [o2] "v"(orow + 32 * NL), [o3] "v"(orow + 48 * NL), [o4] "v"(orow + 64 * NL)
                             : QUANT_CLOBBERS, "scc", "memory");
            }
        }
        asm volatile("ag_e0:");
    } else if (wave == 4) {
        const u32 nbh = lane >> 5, r8 = (lane >> 2) & 7, pos = lane & 3;
        const u32 v0 = r8 * (NL * 2) + tn * 256 + (2 * pos + nbh) * 16;
        const u32 v1 = r8 * (NL * 2) + tn * 256 + (2 * (pos ^ 1) + nbh) * 16;
        u32x4 rs;
        rs[0] = (u32)(u64)P.w; rs[1] = (u32)((u64)P.w >> 32); rs[2] = (u32)GK * NL * 2; rs[3] = 0x00020000u;
        u32 lb = L_RING, freed = 0;
        for (u32 s = 0; s < 128; ++s) {
            if (s == (b < EARLY_WG ? 0 : RING - 2)) {
                while (rfl(lds_ld(L_ECNT)) < 4) __builtin_amdgcn_s_sleep(2);
            }
            if (s == 0 && two) { while (rfl(lds_ld(L_ECNT2)) < 4) __builtin_amdgcn_s_sleep(2); }
            if (s >= RING + freed) {
                const u32 need = s - RING + 1;
                const u64 t0 = rt();
                for (u32 it = 0;; ++it) {
                    u32 cv = lds_ld(L_CONS + (lane & 3) * 256);
                    cv = min(cv, (u32)__builtin_amdgcn_update_dpp((int)cv, (int)cv, 0xb1, 0xf, 0xf, false));
                    cv = min(cv, (u32)__builtin_amdgcn_update_dpp((int)cv, (int)cv, 0x4e, 0xf, 0xf, false));
                    freed = max(rfl(cv), 3u) - 3;
                    if (freed >= need) break;
                    if (it == 0) { waitvm<0>(); if (lane < 4) lds_st(L_PAIR + lane * 8 + 4, s); }
                    __builtin_amdgcn_s_sleep(2);
                    if ((it & 63) == 63) {
                        if (rfl(lds_ld(L_ABORT)) != 0) { freed = 128; break; }
                        if (rt() - t0 > 2 * DEADLINE) { st_agent(ws + OFF_TMO, call); lds_st(L_ABORT, 1); freed = 128; break; }
                    }
                }
            }
            const u32 sb = s * 32 * (NL * 2);
#pragma unroll
            for (int i = 0; i < 8; ++i) {
                u32 keep;
                asm volatile("s_mov_b32 %0, m0\n\ts_mov_b32 m0, %3\n\ts_nop 0\n\tbuffer_load_dwordx4 %1, %2, %4 offen lds\n\ts_mov_b32 m0, %0"
                             : "=&s"(keep) : "v"((i & 1) ? v1 : v0), "s"(rs), "s"(lb + i * 1024), "s"(sb + (i & 3) * 8 * (NL * 2) + (i >> 2) * 128) : "memory");
            }
            lb = lb + SLOT == L_RING + RING * SLOT ? L_RING : lb + SLOT;
            if (s >= 3) {
                waitvm<24>();
                if (lane < 4) asm volatile("ds_max_u32 %0, %1" :: "v"(L_PAIR + lane * 8 + 4), "v"(s - 2) : "memory");
            }
        }
        waitvm<0>();
        if (lane < 4) lds_st(L_PAIR + lane * 8 + 4, 1000);
        asm volatile("ag_e1:");
    } else if (wave == 5) {
        const u32 col = min(tm * 5 + min(lane & 7, 4u), 63u), row = lane >> 3;
        u32 kc = 0;
        const u64 t0 = rt();
        for (u32 it = 0;; ++it) {
            bool ok = true;
            const bool rawp = kc < RAWKB;
            if ((lane & 7) < 5 && row < (rawp ? 4 : POLL_ROWS)) {
                const char* fp = ws + (rawp ? OFF_RAWF + (col * 4 + row) * 128 : OFF_XFLAG + (min(kc + row, 127u) * 64 + col) * 128);
                const u32 v = poll32(fp);
                ok = (int)(v - call) >= 0;
            }
            const u64 mk = __ballot(ok);
            const u32 adv = rawp ? (mk == ~0ull ? RAWKB : 0) : mk == ~0ull ? POLL_ROWS : (u32)__builtin_ctzll(~mk) >> 3;
            if (adv) {
                kc += adv;
                if (lane < 4) lds_st(L_PAIR + lane * 8, kc >= 128 ? 129 : kc);
                if (kc >= 128) break;
                for (u32 k = 0; k < 4096; ++k) {
                    if (__any(lds_ld(L_CONS + (lane & 3) * 256) + (POLL_THROTTLE - 3) >= kc || kc <= POLL_THROTTLE)) break;
                    __builtin_amdgcn_s_sleep(2);
                }
                continue;
            }
            if ((it & 15) == 15) {
                const bool dead = rfl(ld_agent(ws + OFF_TMO)) == call || rfl(lds_ld(L_ABORT)) != 0 || rt() - t0 > DEADLINE;
                if (dead) { st_agent(ws + OFF_TMO, call); lds_st(L_ABORT, 1); if (lane < 4) lds_st(L_PAIR + lane * 8, 129); break; }
            }
            __builtin_amdgcn_s_sleep(POLL_SLEEP);
        }
        const char* gp = lane < 10 ? ws + OFF_AMAXG + lane * 128 : ws + OFF_TMO;
        u32 l7 = lane & 7; asm volatile("" : "+v"(l7));
        char* pm = P.rot[l7] + OFF_AMAXG + 8 * 128;
        asm volatile("" : "+v"(pm));
        u32 own = 0, hold = 1 + (tn & 7);
        bool fin = false;
        const u64 t1 = rt();
        u32 sent = 0, lastg = 0xffffffffu;
        float inv = 0.f;
        for (u32 it = 0; sent < 2; ++it) {
            u64 v = 0;
            if (lane < 11) v = ld_sys64(gp);
            const bool in = (u32)(v >> 32) == call && lane < 10;
            if (!fin) {
                u32x2 mc;
                for (u32 k = 0; k < PRE_SPIN; ++k) {
                    asm volatile("ds_read2_b32 %0, %1 offset1:1\n\ts_waitcnt lgkmcnt(0)" : "=&v"(mc) : "v"((u32)L_MAX) : "memory");
                    if (rfl(mc[1]) >= 16) { fin = true; own = mc[0]; break; }
                    __builtin_amdgcn_s_sleep(1);
                }
                if (!fin && (it & 0xffff) != 0xffff) continue;
                if (!fin && rt() - t1 <= DEADLINE) continue;
            }
            const u32 mk = (u32)__ballot(in);
            if (hold && ((mk >> 8 & 1) || --hold == 0)) {
                hold = 0;
                if (!__any(in && (u32)v >= own) && lane < NRANK)
                    asm volatile("global_atomic_umax_x2 %0, %1, off sc1" :: "v"(pm), "v"(((u64)call << 32) | own) : "memory");
            }
            u32 state = (mk & 255) == 255 ? 2 : (mk & ((1u << 9) | (1u << P.rank))) ? 1 : 0;
            if (__any(lane == 10 && (u32)v == call)) state = 3;
            if (state < 2 && (it & 15) == 15 && rt() - t1 > DEADLINE) { st_agent(ws + OFF_TMO, call); state = 3; }
            u32 gm = lastg;
            if (state && (lastg == 0xffffffffu || __any(in && (u32)v > lastg))) {
                gm = in ? max((u32)v, own) : own;
                gm = max(gm, (u32)__builtin_amdgcn_update_dpp(0, (int)gm, 0x111, 0xf, 0xf, false));
                gm = max(gm, (u32)__builtin_amdgcn_update_dpp(0, (int)gm, 0x112, 0xf, 0xf, false));
                gm = max(gm, (u32)__builtin_amdgcn_update_dpp(0, (int)gm, 0x114, 0xf, 0xf, false));
                gm = max(gm, (u32)__builtin_amdgcn_update_dpp(0, (int)gm, 0x118, 0xf, 0xf, false));
                gm = (u32)__builtin_amdgcn_readlane((int)gm, 15);
            }
            if (state > sent || (state && gm != lastg)) {
                if (gm != lastg) { const float amax = __uint_as_float(gm); inv = amax > 0.f ? 127.0f / amax : 0.f; lastg = gm; }
                if (lane == 0) asm volatile("ds_write_b64 %0, %1" :: "v"((u32)L_RES), "v"(u32x2{state, __float_as_uint(inv)}) : "memory");
                sent = state;
            } else if (!state) __builtin_amdgcn_s_sleep(AMAX_SLEEP);
        }
    }
}

extern "C" void kernel_launch(int rank, int nranks, ncclComm_t comm, hipStream_t stream,
                              void* const* d_in, const int* in_sizes, int n_in,
                              void* d_out, int out_size,
                              const int* in_dtypes, int out_dtype,
                              void* const* d_out_all,
                              void* const* d_ws_all, size_t ws_bytes) {
    if (nranks != NRANK || n_in < 2 || ws_bytes < WS_NEED || (unsigned)rank >= (unsigned)NRANK ||
        in_sizes[0] != MS * GK || in_sizes[1] != GK * NL || out_size != GM * NL) {
        fprintf(stderr, "unexpected problem geometry\n");
        return;
    }
    Params P;
    P.x = (const char*)d_in[0]; P.w = (const char*)d_in[1]; P.out = (char*)d_out;
    for (int i = 0; i < NRANK; ++i) P.rot[i] = (char*)d_ws_all[(rank + i) % NRANK];
    P.rank = rank; P.pad = 0;
    hipLaunchKernelGGL(ag_gemm_kernel, dim3(NWG), dim3(512), 0, stream, P);
    HIP_CHECK(hipGetLastError());
}
```
